```python
import numpy as np

kernel_name = "gpu_ag_gemm_m4096_k4096_n4096_f32_q8"


ABSMAX_RTOL = 0.02

COMPUTE_DTYPE = "float32"
EPILOGUE = "quant_int8"
IN_QUANT = None
OUT_QUANT = "int8"
IN_DTYPES = {"x": "float32", "w_mat": "float32"}
WORLD_SIZE = 8

def setup_inputs():

    x = np.random.randn(4096, 4096).astype(np.float32)
    w_mat = (np.random.randn(4096, 4096) / np.sqrt(4096)).astype(np.float32)
    return {"x": x, "w_mat": w_mat}

def _quant_dequant(y, amax):
    scale = amax / 127.0
    q = np.clip(np.round(y / scale), -127.0, 127.0)
    return (q * scale).astype(np.float32)
def _epilogue(y):
    return _quant_dequant(y, np.abs(y).max())

def reference(x, w_mat):
    y = (x @ w_mat).astype(np.float32)
    return _epilogue(y)
```

```cpp
#include <hip/hip_runtime.h>
#include <rccl/rccl.h>
#include <cstdio>

#define HIP_CHECK(expr) do { hipError_t _e = (expr); if (_e != hipSuccess) { \
    fprintf(stderr, "HIP error %s at %s:%d\n", hipGetErrorString(_e), __FILE__, __LINE__); \
    return; } } while (0)

typedef unsigned u32;
typedef unsigned long long u64;
typedef u32 u32x2 __attribute__((ext_vector_type(2)));
typedef u32 u32x4 __attribute__((ext_vector_type(4)));
typedef float f32x2 __attribute__((ext_vector_type(2)));
typedef float f32x4 __attribute__((ext_vector_type(4)));
typedef __bf16 bf16x2 __attribute__((ext_vector_type(2)));
typedef __bf16 bf16x8 __attribute__((ext_vector_type(8)));
typedef unsigned short u16x2 __attribute__((ext_vector_type(2)));
typedef __amdgpu_buffer_rsrc_t rsrc_t;
#define LAS __attribute__((address_space(3)))
#define GAS __attribute__((address_space(1)))

constexpr int NR = 8;
constexpr int KDIM = 4096, SHARD = 512;
constexpr int NKS = 128;
constexpr int NBLK = 48;
constexpr u32 UNIT = 4096;
#define GSTRIDE 8
#define PW 40
constexpr u32 OFF_CALL = 0, OFF_TMO = 128;
constexpr u32 OFF_AMAX = 4096;
constexpr u32 OFF_DONE = 16384;
constexpr u32 OFF_GRAN = 65536;
constexpr u32 OFF_STAGE = 1u << 20;
constexpr u32 WS_USED = OFF_STAGE + NKS * NBLK * UNIT;
static_assert(OFF_GRAN + NKS * NBLK * GSTRIDE <= OFF_STAGE, "granules fit");
constexpr u32 OOB = 0x80000000u;
constexpr int SYS = 17;
constexpr u64 DEADLINE = 40000000ull;

constexpr int NWAVE = 12, NTHREAD = NWAVE * 64;
constexpr int NSLOT = 8, SLOT = 16384;
constexpr int L_FULL = NSLOT * SLOT;
constexpr int L_PROG = L_FULL + 32;
constexpr int L_WMAX = L_PROG + 16;
constexpr int L_GMAX = L_WMAX + 16;
constexpr int L_ABORT = L_GMAX + 32;
constexpr int LDS_BYTES = L_ABORT + 32;

struct Params {
    const float* x;
    const float* w;
    char* ws[NR];
    signed char* out[NR];
    int rank;
    int pad;
};

__device__ __forceinline__ u64 rt() { return __builtin_amdgcn_s_memrealtime(); }
__device__ __forceinline__ rsrc_t mk(const void* p, u32 bytes) {
    return __builtin_amdgcn_make_buffer_rsrc(const_cast<void*>(p), 0, bytes, 0x00020000);
}
__device__ __forceinline__ u32 uni(u32 v) { return __builtin_amdgcn_readfirstlane(v); }
__device__ __forceinline__ u32 cvt2(float a, float b) {
    f32x2 v = {a, b};
    return __builtin_bit_cast(u32, __builtin_convertvector(v, bf16x2));
}
__device__ __forceinline__ u32 bfi(u32 m, u32 a, u32 b) { return (m & a) | (~m & b); }
__device__ __forceinline__ u32 wave_max_u32(u32 v) {
    v = max(v, (u32)__builtin_amdgcn_update_dpp(0, (int)v, 0x111, 0xf, 0xf, true));
    v = max(v, (u32)__builtin_amdgcn_update_dpp(0, (int)v, 0x112, 0xf, 0xf, true));
    v = max(v, (u32)__builtin_amdgcn_update_dpp(0, (int)v, 0x114, 0xf, 0xf, true));
    v = max(v, (u32)__builtin_amdgcn_update_dpp(0, (int)v, 0x118, 0xf, 0xf, true));
    v = max(v, (u32)__builtin_amdgcn_update_dpp(0, (int)v, 0x142, 0xa, 0xf, false));
    v = max(v, (u32)__builtin_amdgcn_update_dpp(0, (int)v, 0x143, 0xc, 0xf, false));
    return (u32)__builtin_amdgcn_readlane((int)v, 63);
}
#define WAIT_VM(n) asm volatile("s_waitcnt vmcnt(" #n ")" ::: "memory")
#define WAIT_LGKM0() asm volatile("s_waitcnt lgkmcnt(0)" ::: "memory")
#define CBAR() asm volatile("" ::: "memory")
constexpr u32 MSK = 0x87878787u;

template <bool ISX>
__device__ __forceinline__ void push_loop(rsrc_t src, u32 svoff, rsrc_t dst, rsrc_t loc, bool local, u32 dblk,
                                          u32 lblk, int ks0, u32 call, int lane) {
    constexpr int KSTEP = PW / 5, NIT = NKS / KSTEP;
    f32x4 f[8];
    auto load = [&](int ks) {
#pragma unroll
        for (int j = 0; j < 8; ++j) {
            const u32 so = ISX ? (u32)ks * 128u + (u32)(j >> 1) * (16u * KDIM * 4u) : (u32)(ks * 32 + j) * (SHARD * 4u);
            f[j] = __builtin_bit_cast(f32x4, __builtin_amdgcn_raw_buffer_load_b128(src, svoff + (ISX ? (j & 1) * 16 : 0), so, 0));
        }
    };
    load(ks0);
#pragma unroll
    for (int j = 0; j < 10; ++j) __builtin_amdgcn_raw_buffer_store_b32(0u, loc, OOB + 4 * j, 0, 0);
    const u32 lo16 = (u32)lane * 16u;
    const u32 lvo = local ? lo16 : OOB;
    u32 pg = 0, pgd = OOB, pgl = OOB;
    int ks = ks0;
#pragma unroll 1
    for (int it = 0; it < NIT; ++it) {
        u32 pk[16];
#pragma unroll
        for (int g = 0; g < 4; ++g)
#pragma unroll
            for (int i = 0; i < 4; ++i) {
                float a, b;
                if (ISX) { a = f[2 * g + (i >> 1)][(i & 1) * 2]; b = f[2 * g + (i >> 1)][(i & 1) * 2 + 1]; }
                else     { a = f[2 * i][g]; b = f[2 * i + 1][g]; }
                pk[g * 4 + i] = cvt2(a, b);
            }
        load(ks + KSTEP);
        u16x2 mx = __builtin_bit_cast(u16x2, pk[0] & 0x7fff7fffu);
#pragma unroll
        for (int i = 1; i < 16; ++i) mx = __builtin_elementwise_max(mx, __builtin_bit_cast(u16x2, pk[i] & 0x7fff7fffu));
        const u32 top = wave_max_u32(max((u32)mx[0], (u32)mx[1]) >> 8);
        const u32 K4 = (0x80u + top) * 0x01010101u;
        u32x4 lo0, lo1, nib, ext;
#pragma unroll
        for (int g = 0; g < 4; ++g) {
            const u32 L0 = __builtin_amdgcn_perm(pk[g * 4 + 1], pk[g * 4 + 0], 0x06040200u);
            const u32 L1 = __builtin_amdgcn_perm(pk[g * 4 + 3], pk[g * 4 + 2], 0x06040200u);
            const u32 CA = K4 - __builtin_amdgcn_perm(pk[g * 4 + 1], pk[g * 4 + 0], 0x07050301u);
            const u32 CB = K4 - __builtin_amdgcn_perm(pk[g * 4 + 3], pk[g * 4 + 2], 0x07050301u);
            const u32 R = __builtin_amdgcn_alignbit(CB, CB, 28);
            nib[g] = bfi(MSK, CA, R);
            ext[g] = bfi(MSK, R, CA);
            if (g < 2) { lo0[2 * g] = L0; lo0[2 * g + 1] = L1; } else { lo1[2 * g - 4] = L0; lo1[2 * g - 3] = L1; }
        }
        const u64 bal = __ballot((ext[0] | ext[1] | ext[2] | ext[3]) != 0u);
        const u32 m32 = (u32)bal | (u32)(bal >> 32);
        const u32 m16 = (m32 | (m32 >> 16)) & 0xffffu;
        const bool he = (m16 >> (lane & 15)) & 1u;
        const u32 sd = OFF_STAGE + ((u32)ks * NBLK + dblk) * UNIT, sl = OFF_STAGE + ((u32)ks * NBLK + lblk) * UNIT;
        __builtin_amdgcn_raw_buffer_store_b128(lo0, dst, lo16, sd, SYS);
        __builtin_amdgcn_raw_buffer_store_b128(lo1, dst, lo16 + 1024, sd, SYS);
        __builtin_amdgcn_raw_buffer_store_b128(nib, dst, lo16 + 2048, sd, SYS);
        __builtin_amdgcn_raw_buffer_store_b128(ext, dst, he ? lo16 + 3072 : OOB, sd, SYS);
        __builtin_amdgcn_raw_buffer_store_b128(lo0, loc, lvo, sl, SYS);
        __builtin_amdgcn_raw_buffer_store_b128(lo1, loc, lvo + 1024, sl, SYS);
        __builtin_amdgcn_raw_buffer_store_b128(nib, loc, lvo + 2048, sl, SYS);
        __builtin_amdgcn_raw_buffer_store_b128(ext, loc, he ? lvo + 3072 : OOB, sl, SYS);
        WAIT_VM(16);
        const u32x2 gv = {pg, call};
        __builtin_amdgcn_raw_buffer_store_b64(gv, dst, pgd, 0, SYS);
        __builtin_amdgcn_raw_buffer_store_b64(gv, loc, pgl, 0, SYS);
        pg = top | (m16 << 8);
        pgd = lane == 0 ? OFF_GRAN + ((u32)ks * NBLK + dblk) * GSTRIDE : OOB;
        pgl = (lane == 0 && local) ? OFF_GRAN + ((u32)ks * NBLK + lblk) * GSTRIDE : OOB;
        ks += KSTEP;
    }
    WAIT_VM(0);
    const u32x2 gv = {pg, call};
    __builtin_amdgcn_raw_buffer_store_b64(gv, dst, pgd, 0, SYS);
    __builtin_amdgcn_raw_buffer_store_b64(gv, loc, pgl, 0, SYS);
}

__device__ __forceinline__ void decode_store(u32x4 lo0, u32x4 lo1, u32x4 nib, u32x4 ext, u32 K4, LAS char* dst) {
#pragma unroll
    for (int g = 0; g < 4; ++g) {
        const u32 CA = bfi(MSK, nib[g], ext[g]);
        const u32 R = bfi(MSK, ext[g], nib[g]);
        const u32 CB = __builtin_amdgcn_alignbit(R, R, 4);
        const u32 H0 = K4 - CA, H1 = K4 - CB;
        const u32 L0 = g < 2 ? lo0[2 * g] : lo1[2 * g - 4], L1 = g < 2 ? lo0[2 * g + 1] : lo1[2 * g - 3];
        u32x4 d;
        d[0] = __builtin_amdgcn_perm(H0, L0, 0x05010400u);
        d[1] = __builtin_amdgcn_perm(H0, L0, 0x07030602u);
        d[2] = __builtin_amdgcn_perm(H1, L1, 0x05010400u);
        d[3] = __builtin_amdgcn_perm(H1, L1, 0x07030602u);
        *(LAS u32x4*)(dst + g * 1024) = d;
    }
}

__global__ void __launch_bounds__(NTHREAD, 3) ag_gemm_q8(Params p) {
    extern __shared__ __attribute__((aligned(16))) char smem[];
    LAS char* L = (LAS char*)smem;
    const int tid = threadIdx.x, lane = tid & 63;
    const int wave = (int)uni((u32)tid >> 6);
    const int b = blockIdx.x, rank = p.rank & (NR - 1);
    char* const myws = p.ws[rank];
    const rsrc_t rloc = mk(myws, WS_USED);

    if (b < 128) {
        const int pw = wave * 128 + b;
        if (pw < 7 * PW) {
            const u32 call = uni(__hip_atomic_load((GAS u32*)(myws + OFF_CALL), __ATOMIC_RELAXED, __HIP_MEMORY_SCOPE_AGENT)) + 1u;
            const int link = pw % 7, v = pw / 7, t = v % 5, ks0 = v / 5;
            const int dest = (link < 3 ? ((rank & 4) | ((rank + 1 + link) & 3)) : (((rank ^ 4) & 4) | ((rank + link) & 3))) & (NR - 1);
            const rsrc_t rdst = mk(p.ws[dest], WS_USED);
            if (link < 3 || t < 3) {
                const int rb = link < 3 ? t : 5 + t;
                const u32 dblk = link < 3 ? (rank & 3) * 5 + t : 20 + (rank & 3) * 3 + t;
                const bool local = link < 3 && (t % 3) == link;
                const u32 svoff = ((u32)(rb * 64 + (lane & 15)) * KDIM + 8u * (lane >> 4)) * 4u;
                push_loop<true>(mk(p.x, SHARD * KDIM * 4), svoff, rdst, rloc, local, dblk, dblk, ks0, call, lane);
            } else {
                const int h = t - 3, cb = (dest & 3) * 2 + h;
                const u32 dblk = 40 + (rank & 3) * 2 + h;
                const u32 svoff = (8u * (lane >> 4) * SHARD + cb * 64 + 4u * (lane & 15)) * 4u;
                push_loop<false>(mk(p.w, SHARD * KDIM * 4), svoff, rdst, rloc, true, dblk, 32 + cb, ks0, call, lane);
            }
        } else if (b == 0 && wave == NWAVE - 1) {
            const u32 call = uni(__hip_atomic_load((GAS u32*)(myws + OFF_CALL), __ATOMIC_RELAXED, __HIP_MEMORY_SCOPE_AGENT)) + 1u;
            const u64 t0 = rt();
            bool bad = false;
            while (true) {
                const u32 d = __builtin_amdgcn_raw_buffer_load_b32(rloc, OFF_DONE + lane * 4, 0, SYS);
                if (__all(d >= call)) break;
                if (rt() - t0 > DEADLINE) { bad = true; break; }
                __builtin_amdgcn_s_sleep(2);
            }
            if (lane == 0) {
                if (bad) {
                    __hip_atomic_store((GAS u32*)(myws + OFF_TMO), call, __ATOMIC_RELAXED, __HIP_MEMORY_SCOPE_AGENT);
                    __hip_atomic_store((GAS signed char*)(p.out[rank] + (size_t)rank * SHARD * SHARD), (signed char)-128, __ATOMIC_RELAXED, __HIP_MEMORY_SCOPE_SYSTEM);
                }
                __hip_atomic_store((GAS u32*)(myws + OFF_CALL), call, __ATOMIC_RELAXED, __HIP_MEMORY_SCOPE_AGENT);
            }
        }
        return;
    }

    const int xl = b & 7, li = (b >> 3) - 16;
    const int ti = 4 * (xl >> 1) + (li & 3);
    const int tj = 2 * (xl & 1) + ((li >> 2) & 1) + 4 * (li >> 3);
    const int gw = ti * 8 + tj;
    volatile LAS u32* FULL = (volatile LAS u32*)(L + L_FULL);
    volatile LAS u32* PROG = (volatile LAS u32*)(L + L_PROG);
    volatile LAS u32* WMAX = (volatile LAS u32*)(L + L_WMAX);
    volatile LAS u32* GMAX = (volatile LAS u32*)(L + L_GMAX);
    volatile LAS u32* ABORT = (volatile LAS u32*)(L + L_ABORT);
    if (tid < (L_ABORT + 32 - L_FULL) / 4) ((volatile LAS u32*)(L + L_FULL))[tid] = 0u;
    u32 call = 0;
    if (wave >= 4) call = uni(__hip_atomic_load((GAS u32*)(myws + OFF_CALL), __ATOMIC_RELAXED, __HIP_MEMORY_SCOPE_AGENT)) + 1u;
    __syncthreads();

    f32x4 acc[4][4];
    if (wave < 4) {
        __builtin_amdgcn_s_setprio(2);
        const int wm = wave >> 1, wn = wave & 1;
        LAS char* pa = L + wm * 4096 + lane * 16;
        LAS char* pb = L + 8192 + wn * 4096 + lane * 16;
#pragma unroll
        for (int i = 0; i < 4; ++i)
#pragma unroll
            for (int j = 0; j < 4; ++j) acc[i][j] = f32x4{0.f, 0.f, 0.f, 0.f};
        bf16x8 A0[4], B0[4], A1[4], B1[4];
        auto rd = [&](bf16x8* A, bf16x8* B, int s) {
            const int o = (s & 7) * SLOT;
#pragma unroll
            for (int g = 0; g < 4; ++g) { A[g] = *(LAS bf16x8*)(pa + o + g * 1024); B[g] = *(LAS bf16x8*)(pb + o + g * 1024); }
        };
        auto mm = [&](bf16x8* A, bf16x8* B) {
#pragma unroll
            for (int i = 0; i < 4; ++i)
#pragma unroll
                for (int j = 0; j < 4; ++j) acc[i][j] = __builtin_amdgcn_mfma_f32_16x16x32_bf16(B[j], A[i], acc[i][j], 0, 0, 0);
        };
        auto wait_full = [&](u32 seen, int s) {
            while (seen < (u32)s + 1u) seen = FULL[s & 7];
            CBAR();
        };
        wait_full(0u, 0);
        rd(A0, B0, 0);
        u32 fl = FULL[1];
#pragma unroll 1
        for (int s = 0; s < NKS; s += 2) {
            wait_full(fl, s + 1);
            WAIT_LGKM0();
            PROG[wave] = (u32)s + 1u;
            rd(A1, B1, s + 1);
            fl = FULL[(s + 2) & 7];
            __builtin_amdgcn_sched_barrier(0);
            mm(A0, B0);
            __builtin_amdgcn_sched_barrier(0);
            if (s + 2 < NKS) wait_full(fl, s + 2);
            WAIT_LGKM0();
            PROG[wave] = (u32)s + 2u;
            rd(A0, B0, s + 2);
            fl = FULL[(s + 3) & 7];
            __builtin_amdgcn_sched_barrier(0);
            mm(A1, B1);
            __builtin_amdgcn_sched_barrier(0);
        }
        float m = 0.f;
#pragma unroll
        for (int i = 0; i < 4; ++i)
#pragma unroll
            for (int j = 0; j < 4; ++j)
#pragma unroll
                for (int r = 0; r < 4; ++r) m = fmaxf(m, fabsf(acc[i][j][r]));
        const u32 wmx = wave_max_u32(__builtin_bit_cast(u32, m));
        WMAX[wave] = wmx;
    } else {
        const int d = wave - 4;
        const u32 gidx = lane < 2 ? 2 * ti + lane : 32 + 2 * tj + (lane - 2);
        const u32 gvo = lane < 4 ? OFF_GRAN + gidx * GSTRIDE : OOB;
        const u32 lo16 = (u32)lane * 16u;
        LAS char* dst = L + d * SLOT + lane * 16;
        u32x2 gr = __builtin_amdgcn_raw_buffer_load_b64(rloc, gvo, (u32)d * (NBLK * GSTRIDE), SYS);
#pragma unroll 1
        for (int s = d; s < NKS; s += 8) {
            if (((u32)__ballot(gr[1] >= call) & 15u) != 15u) {
                const u64 t0 = rt();
                while (true) {
                    gr = __builtin_amdgcn_raw_buffer_load_b64(rloc, gvo, (u32)s * (NBLK * GSTRIDE), SYS);
                    if (((u32)__ballot(gr[1] >= call) & 15u) == 15u) break;
                    if (*ABORT != 0u) break;
                    if (rt() - t0 > DEADLINE) {
                        *ABORT = 1u;
                        if (lane == 0) __hip_atomic_store((GAS u32*)(myws + OFF_TMO), call, __ATOMIC_RELAXED, __HIP_MEMORY_SCOPE_AGENT);
                        break;
                    }
                    __builtin_amdgcn_s_sleep(1);
                }
            }
            CBAR();
            u32 gl[4];
#pragma unroll
            for (int u = 0; u < 4; ++u) gl[u] = (u32)__builtin_amdgcn_readlane((int)gr[0], u);
            gr = __builtin_amdgcn_raw_buffer_load_b64(rloc, s + 8 < NKS ? gvo : OOB, (u32)(s + 8) * (NBLK * GSTRIDE), SYS);
            u32x4 q[4][4];
#pragma unroll
            for (int u = 0; u < 4; ++u) {
                const u32 blk = u < 2 ? 2 * ti + u : 32 + 2 * tj + (u - 2);
                const u32 so = OFF_STAGE + ((u32)s * NBLK + blk) * UNIT;
                const bool he = (gl[u] >> (8 + (lane & 15))) & 1u;
                q[u][0] = __builtin_amdgcn_raw_buffer_load_b128(rloc, lo16, so, SYS);
                q[u][1] = __builtin_amdgcn_raw_buffer_load_b128(rloc, lo16 + 1024, so, SYS);
                q[u][2] = __builtin_amdgcn_raw_buffer_load_b128(rloc, lo16 + 2048, so, SYS);
                q[u][3] = __builtin_amdgcn_raw_buffer_load_b128(rloc, he ? lo16 + 3072 : OOB, so, SYS);
            }
            if (s >= NSLOT) {
                const u32 need = (u32)(s - NSLOT + 1);
                while (true) {
                    const u32x4 pr = *(volatile LAS u32x4*)(L + L_PROG);
                    if (min(min(pr[0], pr[1]), min(pr[2], pr[3])) >= need) break;
                    __builtin_amdgcn_s_sleep(1);
                }
            }
            CBAR();
#pragma unroll
            for (int u = 0; u < 4; ++u) decode_store(q[u][0], q[u][1], q[u][2], q[u][3], (0x80u + (gl[u] & 0x7fu)) * 0x01010101u, dst + u * 4096);
            WAIT_LGKM0();
            FULL[d] = (u32)s + 1u;
        }
    }
    __syncthreads();
    if (wave >= 4) {
        const int d = wave - 4;
        if (d == 0 && lane < NR) {
            const u32x4 wm4 = *(volatile LAS u32x4*)(L + L_WMAX);
            const u32 m = max(max(wm4[0], wm4[1]), max(wm4[2], wm4[3]));
            __hip_atomic_store((GAS u64*)(p.ws[lane] + OFF_AMAX + (rank * 128 + gw) * 8), ((u64)call << 32) | m, __ATOMIC_RELAXED, __HIP_MEMORY_SCOPE_SYSTEM);
        }
        const u64 t0 = rt();
        u32x4 g2;
        while (true) {
            g2 = __builtin_amdgcn_raw_buffer_load_b128(rloc, OFF_AMAX + d * 1024 + lane * 16, 0, SYS);
            if (__all(g2[1] >= call && g2[3] >= call)) break;
            if (*ABORT != 0u) break;
            if (rt() - t0 > DEADLINE) {
                *ABORT = 1u;
                if (lane == 0) __hip_atomic_store((GAS u32*)(myws + OFF_TMO), call, __ATOMIC_RELAXED, __HIP_MEMORY_SCOPE_AGENT);
                break;
            }
            __builtin_amdgcn_s_sleep(1);
        }
        GMAX[d] = wave_max_u32(max(g2[0], g2[2]));
    }
    __syncthreads();
    if (wave < 4) {
        const u32x4 ga = *(volatile LAS u32x4*)(L + L_GMAX), gb = *(volatile LAS u32x4*)(L + L_GMAX + 16);
        const u32 mb = max(max(max(ga[0], ga[1]), max(ga[2], ga[3])), max(max(gb[0], gb[1]), max(gb[2], gb[3])));
        const float amax = __builtin_bit_cast(float, mb);
        const float scale = amax / 127.0f;
        const float inv = scale > 0.f ? 1.0f / scale : 0.f;
        const bool bad = *ABORT != 0u;
        const int wm = wave >> 1, wn = wave & 1;
#pragma unroll
        for (int i = 0; i < 4; ++i) {
            u32x4 o;
#pragma unroll
            for (int r = 0; r < 4; ++r) {
                u32 qb[4];
#pragma unroll
                for (int j = 0; j < 4; ++j) {
                    const float y = __builtin_amdgcn_fmed3f(acc[i][j][r] * inv, -127.0f, 127.0f);
                    qb[j] = __builtin_bit_cast(u32, y + 12582912.0f);
                }
                o[r] = __builtin_amdgcn_perm(__builtin_amdgcn_perm(qb[3], qb[2], 0x0c0c0400u), __builtin_amdgcn_perm(qb[1], qb[0], 0x0c0c0400u), 0x05040100u);
                if (bad) o[r] = 0x80808080u;
            }
            const int row = wm * 64 + i * 16 + (lane & 15), ch = wn * 4 + (lane >> 4);
            *(LAS u32x4*)(L + row * 128 + ((ch ^ (row & 7)) * 16)) = o;
        }
    }
    __syncthreads();
    const int owner = (tj < 4 ? rank : (((rank ^ 4) & 4) | (tj - 4))) & (NR - 1);
    if (wave < 8) {
        const rsrc_t rout = mk(p.out[owner], KDIM * SHARD);
        const u32 col = tj < 4 ? tj * 128 : (rank & 3) * 128;
        const int blk = 2 * ti + (wave >> 2);
        const int srcr = blk < 20 ? ((rank & 4) | (blk / 5)) : (((rank ^ 4) & 4) | ((blk - 20) / 3));
        const int rb = blk < 20 ? blk % 5 : 5 + (blk - 20) % 3;
#pragma unroll
        for (int h = 0; h < 2; ++h) {
            const int row = wave * 16 + h * 8 + (lane >> 3), ch = lane & 7;
            const u32x4 o = *(LAS u32x4*)(L + row * 128 + ((ch ^ (row & 7)) * 16));
            const u32 grow = (u32)(srcr * SHARD + rb * 64 + (row & 63));
            __builtin_amdgcn_raw_buffer_store_b128(o, rout, grow * SHARD + col + ch * 16, 0, SYS);
        }
        WAIT_VM(0);
    }
    __syncthreads();
    if (tj >= 4 && wave == 4 && lane == 0)
        __hip_atomic_store((GAS u32*)(p.ws[owner] + OFF_DONE + ((rank & 3) * 16 + ti) * 4), call, __ATOMIC_RELAXED, __HIP_MEMORY_SCOPE_SYSTEM);
}

extern "C" void kernel_launch(int rank, int nranks, ncclComm_t comm, hipStream_t stream,
                              void* const* d_in, const int* in_sizes, int n_in,
                              void* d_out, int out_size, const int* in_dtypes, int out_dtype,
                              void* const* d_out_all, void* const* d_ws_all, size_t ws_bytes) {
    (void)comm; (void)in_dtypes; (void)out_dtype; (void)d_out;
    if (nranks != NR || (unsigned)rank >= (unsigned)NR || n_in < 2 || in_sizes[0] != SHARD * KDIM || in_sizes[1] != SHARD * KDIM ||
        out_size != KDIM * SHARD || ws_bytes < (size_t)WS_USED) {
        fprintf(stderr, "unexpected problem shape\n");
        return;
    }
    Params p;
    p.x = (const float*)d_in[0];
    p.w = (const float*)d_in[1];
    for (int r = 0; r < NR; ++r) { p.ws[r] = (char*)d_ws_all[r]; p.out[r] = (signed char*)d_out_all[r]; }
    p.rank = rank;
    p.pad = 0;
    HIP_CHECK(hipFuncSetAttribute((const void*)ag_gemm_q8, hipFuncAttributeMaxDynamicSharedMemorySize, LDS_BYTES));
    hipLaunchKernelGGL(ag_gemm_q8, dim3(256), dim3(NTHREAD), LDS_BYTES, stream, p);
    HIP_CHECK(hipGetLastError());
}
```

```cpp
#include <hip/hip_runtime.h>
#include <rccl/rccl.h>
#include <cstdio>

#define HIP_CHECK(expr) do { hipError_t _e = (expr); if (_e != hipSuccess) { \
    fprintf(stderr, "HIP error %s at %s:%d\n", hipGetErrorString(_e), __FILE__, __LINE__); \
    return; } } while (0)

typedef unsigned u32;
typedef unsigned long long u64;
typedef u32 u32x2 __attribute__((ext_vector_type(2)));
typedef u32 u32x4 __attribute__((ext_vector_type(4)));
typedef float f32x2 __attribute__((ext_vector_type(2)));
typedef float f32x4 __attribute__((ext_vector_type(4)));
typedef float f32x16 __attribute__((ext_vector_type(16)));
typedef __bf16 bf16x2 __attribute__((ext_vector_type(2)));
typedef __bf16 bf16x8 __attribute__((ext_vector_type(8)));
typedef unsigned short u16x2 __attribute__((ext_vector_type(2)));
typedef __amdgpu_buffer_rsrc_t rsrc_t;
#define LAS __attribute__((address_space(3)))
#define GAS __attribute__((address_space(1)))

constexpr int NR = 8;
constexpr int KDIM = 4096, SHARD = 512;
constexpr int NKS = 128;
constexpr int NBLK = 48;
constexpr u32 UNIT = 4096;
#define GSTRIDE 8
#define NPW 80
constexpr int NUNIT = NKS * 16;
constexpr u32 OFF_CALL = 0, OFF_TMO = 128;
constexpr u32 OFF_TICK = 1024;
constexpr u32 OFF_DONE = 4096;
constexpr u32 OFF_AMAX = 16384;
constexpr u32 OFF_GRAN = 65536;
constexpr u32 OFF_STAGE = 1u << 20;
constexpr u32 WS_USED = OFF_STAGE + NKS * NBLK * UNIT;
static_assert(OFF_GRAN + NKS * NBLK * GSTRIDE <= OFF_STAGE, "granules fit");
constexpr u32 OOB = 0x80000000u;
constexpr int SYS = 17;
constexpr u64 DEADLINE = 40000000ull;

constexpr int NWAVE = 12, NTHREAD = NWAVE * 64;
constexpr int NSLOT = 8, SLOT = 16384;
#define NPWG 128
#define LAYOUT 1
__device__ __forceinline__ u32 unit_off(u32 ks, u32 blk) { return OFF_STAGE + (LAYOUT ? blk * NKS + ks : ks * NBLK + blk) * UNIT; }
constexpr int L_FULL = NSLOT * SLOT;
constexpr int L_PROG = L_FULL + 32;
constexpr int L_WMAX = L_PROG + 16;
constexpr int L_GMAX = L_WMAX + 16;
constexpr int L_ABORT = L_GMAX + 32;
constexpr int LDS_BYTES = L_ABORT + 32;

struct Params {
    const float* x;
    const float* w;
    char* ws[NR];
    signed char* out[NR];
    int rank;
    int pad;
};
#define MODE(bit) 0

#define DEPTH1 0
#define WARM
#define QROT 1
#define GROUPING 0
__host__ __device__ __forceinline__ int phys(int l) { return GROUPING == 0 ? l : (GROUPING == 1 ? ((l & 3) * 2 + (l >> 2)) : (((l & 2) << 1) | ((l >> 1) & 2) | (l & 1))); }
__device__ __forceinline__ u64 rt() { return __builtin_amdgcn_s_memrealtime(); }
__device__ __forceinline__ rsrc_t mk(const void* p, u32 bytes) {
    return __builtin_amdgcn_make_buffer_rsrc(const_cast<void*>(p), 0, bytes, 0x00020000);
}
__device__ __forceinline__ u32 uni(u32 v) { return __builtin_amdgcn_readfirstlane(v); }
__device__ __forceinline__ u32 cvt2(float a, float b) {
    f32x2 v = {a, b};
    return __builtin_bit_cast(u32, __builtin_convertvector(v, bf16x2));
}
__device__ __forceinline__ u32 bfi(u32 m, u32 a, u32 b) { return (m & a) | (~m & b); }
__device__ __forceinline__ u32 wave_max_u32(u32 v) {
    v = max(v, (u32)__builtin_amdgcn_update_dpp(0, (int)v, 0x111, 0xf, 0xf, true));
    v = max(v, (u32)__builtin_amdgcn_update_dpp(0, (int)v, 0x112, 0xf, 0xf, true));
    v = max(v, (u32)__builtin_amdgcn_update_dpp(0, (int)v, 0x114, 0xf, 0xf, true));
    v = max(v, (u32)__builtin_amdgcn_update_dpp(0, (int)v, 0x118, 0xf, 0xf, true));
    v = max(v, (u32)__builtin_amdgcn_update_dpp(0, (int)v, 0x142, 0xa, 0xf, false));
    v = max(v, (u32)__builtin_amdgcn_update_dpp(0, (int)v, 0x143, 0xc, 0xf, false));
    return (u32)__builtin_amdgcn_readlane((int)v, 63);
}
struct Spin {
    u64 t0 = 0;
    u32 n = 0;
    __device__ __forceinline__ bool expired(volatile LAS u32* abort_word) {
        if ((++n & 63u) != 0u) return false;
        if (n == 64u) t0 = rt();
        return (abort_word && *abort_word != 0u) || rt() - t0 > DEADLINE;
    }
};
#define WAIT_VM(n) asm volatile("s_waitcnt vmcnt(" #n ")" ::: "memory")
#define WAIT_LGKM0() asm volatile("s_waitcnt lgkmcnt(0)" ::: "memory")
#define CBAR() asm volatile("" ::: "memory")
#define PSLEEP 1
#define POLL_SLEEP() do { if (PSLEEP) __builtin_amdgcn_s_sleep(PSLEEP); } while (0)
constexpr u32 MSK = 0x87878787u;
#define ILV2() __builtin_amdgcn_sched_group_barrier(0x8, 1, 0); __builtin_amdgcn_sched_group_barrier(0x100, 2, 0)
#define ILV() do { ILV2(); ILV2(); ILV2(); ILV2(); ILV2(); __builtin_amdgcn_sched_group_barrier(0x8, 3, 0); } while (0)
#define MPRIO 0
#define DPRIO 2
#define CYC(x)

__device__ __forceinline__ void push_wave(const Params& p, int rank, int v, rsrc_t loc, int lane
) {
    const u32 xvo = ((u32)(lane >> 2) * KDIM + 4u * (lane & 3)) * 4u;
    const u32 wvo = (4u * (lane & 3) * SHARD + 4u * (lane >> 2)) * 4u;
    f32x4 f[8];
    auto load = [&](u32 j) {
        const u32 ks = j >> 4, u = j & 15u;
        const bool ix = u < 8u;
        const rsrc_t r = mk(ix ? p.x : p.w, SHARD * KDIM * 4);
        const u32 so = ix ? ks * 128u + u * (64u * KDIM * 4u) : ks * (32u * SHARD * 4u) + (u - 8u) * 256u;
        const u32 vo = j < NUNIT ? (ix ? xvo : wvo) : OOB;
#pragma unroll
        for (int i = 0; i < 8; ++i) {
            const u32 d = ix ? (u32)(i >> 1) * (16u * KDIM * 4u) + (i & 1) * 64u : (u32)(16 * (i >> 2) + (i & 3)) * (SHARD * 4u);
            f[i] = __builtin_bit_cast(f32x4, __builtin_amdgcn_raw_buffer_load_b128(r, vo, so + d, 0));
        }
    };
    u32 callv = __builtin_amdgcn_raw_buffer_load_b32(loc, OFF_CALL, 0, 16);
    u32 cur = (u32)v, nxt = (u32)v + NPW;
    load(cur);
    __builtin_amdgcn_s_waitcnt(0x0F70);
    const u32 call = uni(callv) + 1u;
    const u32 tbase = (call - 1u) * (u32)NUNIT - 2u * NPW;
    const u32 lo16 = (u32)lane * 16u;
    const u32 l0 = lane == 0 ? 0u : OOB;
    u32 pg = 0, pgo[4] = {OOB, OOB, OOB, OOB};
    rsrc_t pr[4] = {loc, loc, loc, loc};
#pragma unroll 1
    do {
        const u32 ks = cur >> 4, u = cur & 15u;
        const bool isx = u < 8u;
        rsrc_t dr[4];
        u32 blk[4];
#pragma unroll
        for (int i = 0; i < 4; ++i) {
            int dest;
            if (isx) {
                dest = ((u < 5u ? rank : rank ^ 4) & 4) | ((rank + 1 + i) & 3);
                blk[i] = u < 5u ? (rank & 3) * 5 + u : 15 + (rank & 3) * 3 + u;
            } else {
                dest = i == 0 ? (((rank ^ 4) & 4) | (int)((((u - 8u) >> 1) - QROT * rank) & 3u)) : rank;
                blk[i] = i == 0 ? 40 + (rank & 3) * 2 + (u & 1u) : 24 + u;
            }
            dr[i] = mk(p.ws[dest & (NR - 1)], WS_USED);
        }
        u32 pk[16];
        if (isx) {
#pragma unroll
            for (int i = 0; i < 16; ++i) {
                const int kh = i >> 3, rg = ((i >> 2) & 1) * 2 + (i & 1), e = (i >> 1) & 1;
                pk[i] = cvt2(f[2 * rg + kh][2 * e], f[2 * rg + kh][2 * e + 1]);
            }
        } else {
#pragma unroll
            for (int i = 0; i < 16; ++i) {
                const int kp = (i >> 3) * 2 + ((i >> 1) & 1), c = ((i >> 2) & 1) * 2 + (i & 1);
                pk[i] = cvt2(f[2 * kp][c], f[2 * kp + 1][c]);
            }
        }
        const u32 tk = (u32)__builtin_amdgcn_raw_ptr_buffer_atomic_add_i32(1, loc, l0 + OFF_TICK, 0, 0);
        load(nxt);
        u16x2 mx = __builtin_bit_cast(u16x2, pk[0] & 0x7fff7fffu);
#pragma unroll
        for (int i = 1; i < 16; ++i) mx = __builtin_elementwise_max(mx, __builtin_bit_cast(u16x2, pk[i] & 0x7fff7fffu));
        const u32 top = wave_max_u32(max((u32)mx[0], (u32)mx[1]) >> 8);
        const u32 K4 = (0x80u + top) * 0x01010101u;
        u32x4 lo0, lo1, nib, ext;
#pragma unroll
        for (int g = 0; g < 4; ++g) {
            const u32 L0 = __builtin_amdgcn_perm(pk[g * 4 + 1], pk[g * 4 + 0], 0x06040200u);
            const u32 L1 = __builtin_amdgcn_perm(pk[g * 4 + 3], pk[g * 4 + 2], 0x06040200u);
            const u32 CA = K4 - __builtin_amdgcn_perm(pk[g * 4 + 1], pk[g * 4 + 0], 0x07050301u);
            const u32 CB = K4 - __builtin_amdgcn_perm(pk[g * 4 + 3], pk[g * 4 + 2], 0x07050301u);
            const u32 R = __builtin_amdgcn_alignbit(CB, CB, 28);
            nib[g] = bfi(MSK, CA, R);
            ext[g] = bfi(MSK, R, CA);
            if (g < 2) { lo0[2 * g] = L0; lo0[2 * g + 1] = L1; } else { lo1[2 * g - 4] = L0; lo1[2 * g - 3] = L1; }
        }
        const u64 bal = __ballot((ext[0] | ext[1] | ext[2] | ext[3]) != 0u);
        const u32 m32 = (u32)bal | (u32)(bal >> 32);
        const u32 m16 = (m32 | (m32 >> 16)) & 0xffffu;
        const u32 evo = ((m16 >> (lane & 15)) & 1u) ? lo16 + 3072 : OOB;
        const u32 hvo = isx ? lo16 : OOB, hevo = isx ? evo : OOB;
#pragma unroll
        for (int i = 0; i < 4; ++i) {
            const u32 so = unit_off(ks, blk[i]);
            const u32 vo = i < 2 ? lo16 : hvo;
            __builtin_amdgcn_raw_buffer_store_b128(lo0, dr[i], vo, so, SYS);
            __builtin_amdgcn_raw_buffer_store_b128(lo1, dr[i], vo + 1024, so, SYS);
            __builtin_amdgcn_raw_buffer_store_b128(nib, dr[i], vo + 2048, so, SYS);
            __builtin_amdgcn_raw_buffer_store_b128(ext, dr[i], i < 2 ? evo : hevo, so, SYS);
        }
        WAIT_VM(24);
        const u32x2 gv = {pg, call};
#pragma unroll
        for (int i = 0; i < 4; ++i) {
            __builtin_amdgcn_raw_buffer_store_b64(gv, pr[i], l0 | pgo[i], 0, SYS);
            pr[i] = dr[i];
            pgo[i] = (i < 2 || isx) ? OFF_GRAN + (ks * NBLK + blk[i]) * GSTRIDE : OOB;
        }
        pg = top | (m16 << 8);
        __builtin_amdgcn_s_waitcnt(0x4F74);
        cur = nxt;
        nxt = uni(tk) - tbase;
    } while (cur < (u32)NUNIT);
    WAIT_VM(0);
    const u32x2 gv = {pg, call};
#pragma unroll
    for (int i = 0; i < 4; ++i) __builtin_amdgcn_raw_buffer_store_b64(gv, pr[i], l0 | pgo[i], 0, SYS);
}

template <int OFF>
__device__ __forceinline__ void decode_store(u32x4 lo0, u32x4 lo1, u32x4 nib, u32x4 ext, u32 K4, u32 base) {
#pragma unroll
    for (int g = 0; g < 4; ++g) {
        const u32 CA = bfi(MSK, nib[g], ext[g]);
        const u32 R = bfi(MSK, ext[g], nib[g]);
        const u32 CB = __builtin_amdgcn_alignbit(R, R, 4);
        const u32 H0 = K4 - CA, H1 = K4 - CB;
        const u32 L0 = g < 2 ? lo0[2 * g] : lo1[2 * g - 4], L1 = g < 2 ? lo0[2 * g + 1] : lo1[2 * g - 3];
        const u32 d0 = __builtin_amdgcn_perm(H0, L0, 0x05010400u);
        const u32 d1 = __builtin_amdgcn_perm(H0, L0, 0x07030602u);
        const u32 d2 = __builtin_amdgcn_perm(H1, L1, 0x05010400u);
        const u32 d3 = __builtin_amdgcn_perm(H1, L1, 0x07030602u);
        asm volatile("s_mov_b32 m0, %4\n\ts_nop 0\n\t"
                     "ds_write_addtid_b32 %0 offset:%5\n\tds_write_addtid_b32 %1 offset:%6\n\t"
                     "ds_write_addtid_b32 %2 offset:%7\n\tds_write_addtid_b32 %3 offset:%8"
                     :: "v"(d0), "v"(d1), "v"(d2), "v"(d3), "s"(base), "i"(OFF + g * 1024), "i"(OFF + g * 1024 + 256),
                        "i"(OFF + g * 1024 + 512), "i"(OFF + g * 1024 + 768) : "memory");
    }
}

__global__ void __launch_bounds__(NTHREAD, 3) ag_gemm_q8(Params p) {
    extern __shared__ __attribute__((aligned(16))) char smem[];
    LAS char* L = (LAS char*)smem;
    const int tid = threadIdx.x, lane = tid & 63;
    const int wave = (int)uni((u32)tid >> 6);
    const int b = blockIdx.x, rank = p.rank & (NR - 1);
    char* const myws = p.ws[rank];
    const rsrc_t rloc = mk(myws, WS_USED);

    if (b < 128) {
        const int pw = wave * NPWG + b;
        if (b < NPWG && pw < NPW && !MODE(0)) {
            push_wave(p, rank, pw, rloc, lane);
        } else if (b == 127 && wave == NWAVE - 1) {
            const u32 call = uni(__hip_atomic_load((GAS u32*)(myws + OFF_CALL), __ATOMIC_RELAXED, __HIP_MEMORY_SCOPE_AGENT)) + 1u;
            Spin sp;
            bool bad = false;
            while (true) {
                const u32 d = __builtin_amdgcn_raw_buffer_load_b32(rloc, OFF_DONE + lane * 4, 0, SYS);
                if (__all(d >= call)) break;
                if (sp.expired(nullptr)) { bad = true; break; }
            }
            if (lane == 0) {
                if (bad) {
                    __hip_atomic_store((GAS u32*)(myws + OFF_TMO), call, __ATOMIC_RELAXED, __HIP_MEMORY_SCOPE_AGENT);
                    __hip_atomic_store((GAS signed char*)(p.out[rank] + (size_t)phys(rank) * SHARD * SHARD), (signed char)-128, __ATOMIC_RELAXED, __HIP_MEMORY_SCOPE_SYSTEM);
                }
                __hip_atomic_store((GAS u32*)(myws + OFF_CALL), call, __ATOMIC_RELAXED, __HIP_MEMORY_SCOPE_AGENT);
            }
        }
        return;
    }

    const int xl = b & 7, li = (b >> 3) - 16;
    const int ti = 4 * (xl >> 1) + (li & 3);
    const int tj = 2 * (xl & 1) + ((li >> 2) & 1) + 4 * (li >> 3);
    const int gw = ti * 8 + tj;
    volatile LAS u32* FULL = (volatile LAS u32*)(L + L_FULL);
    volatile LAS u32* PROG = (volatile LAS u32*)(L + L_PROG);
    volatile LAS u32* WMAX = (volatile LAS u32*)(L + L_WMAX);
    volatile LAS u32* GMAX = (volatile LAS u32*)(L + L_GMAX);
    volatile LAS u32* ABORT = (volatile LAS u32*)(L + L_ABORT);
    if (tid < (L_ABORT + 32 - L_FULL) / 4) ((volatile LAS u32*)(L + L_FULL))[tid] = 0u;
    u32 call = 0;
    WAIT_LGKM0();
    __builtin_amdgcn_s_barrier();
    GAS u64* amax_dst = nullptr;
    if (wave >= 4) {
        call = uni(__hip_atomic_load((GAS u32*)(myws + OFF_CALL), __ATOMIC_RELAXED, __HIP_MEMORY_SCOPE_AGENT)) + 1u;
        amax_dst = (GAS u64*)(p.ws[lane & (NR - 1)] + OFF_AMAX + (rank * 128 + gw) * 8);
        asm volatile("" : "+v"(amax_dst));
    }

    f32x16 acc[2][2];
    if (wave < 4) {
        __builtin_amdgcn_s_setprio(MPRIO);
        const int wm = wave >> 1, wn = wave & 1;
        LAS char* pa = L + wm * 4096 + lane * 16;
        LAS char* pb = L + 65536 + wn * 4096 + lane * 16;
#pragma unroll
        for (int i = 0; i < 2; ++i)
#pragma unroll
            for (int j = 0; j < 2; ++j)
#pragma unroll
                for (int r = 0; r < 16; ++r) acc[i][j][r] = 0.f;
        bf16x8 A0[4], B0[4], A1[4], B1[4];
        auto rd = [&](bf16x8* A, bf16x8* B, int s) {
            const int o = (s & 7) * (SLOT / 2);
#pragma unroll
            for (int g = 0; g < 4; ++g) { A[g] = *(LAS bf16x8*)(pa + o + g * 1024); B[g] = *(LAS bf16x8*)(pb + o + g * 1024); }
        };
        auto mm = [&](bf16x8* A, bf16x8* B) {
#pragma unroll
            for (int kh = 0; kh < 2; ++kh)
#pragma unroll
                for (int i = 0; i < 2; ++i)
#pragma unroll
                    for (int j = 0; j < 2; ++j) acc[i][j] = __builtin_amdgcn_mfma_f32_32x32x16_bf16(B[2 * kh + j], A[2 * kh + i], acc[i][j], 0, 0, 0);
        };
        auto wait_full = [&](u32 seen, int s) {
            while (seen < (u32)s + 1u) seen = FULL[s & 7];
            CBAR();
        };
        if (!MODE(2)) wait_full(0u, 0);
        CYC(const u64 c_beg = cyc(); u64 c_spin = 0;)
        rd(A0, B0, 0);
        u32 fl = FULL[1];
#pragma unroll 1
        for (int s = MODE(2) ? NKS : 0; s < NKS; s += 2) {
            CYC(WAIT_LGKM0(); const u64 ca = cyc();)
            wait_full(fl, s + 1);
            CYC(c_spin += cyc() - ca;)
            WAIT_LGKM0();
            PROG[wave] = (u32)s + 1u;
            __builtin_amdgcn_sched_barrier(0);
            rd(A1, B1, s + 1);
            fl = FULL[(s + 2) & 7];
            mm(A0, B0);
            ILV();
            __builtin_amdgcn_sched_barrier(0);
            if (s + 2 < NKS) wait_full(fl, s + 2);
            WAIT_LGKM0();
            PROG[wave] = (u32)s + 2u;
            __builtin_amdgcn_sched_barrier(0);
            rd(A0, B0, s + 2);
            fl = FULL[(s + 3) & 7];
            mm(A1, B1);
            ILV();
            __builtin_amdgcn_sched_barrier(0);
        }
        float m = 0.f;
#pragma unroll
        for (int i = 0; i < 2; ++i)
#pragma unroll
            for (int j = 0; j < 2; ++j)
#pragma unroll
                for (int r = 0; r < 16; ++r) m = fmaxf(m, fabsf(acc[i][j][r]));
        const u32 wmx = wave_max_u32(__builtin_bit_cast(u32, m));
        WMAX[wave] = wmx;
    } else {
        __builtin_amdgcn_s_setprio(DPRIO);
        const int d = wave - 4;
        const u32 gidx = lane < 2 ? 2 * ti + lane : 32 + 2 * tj + (lane - 2);
        const u32 gvo = lane < 4 ? OFF_GRAN + gidx * GSTRIDE : OOB;
        const u32 lo16 = (u32)lane * 16u;
        const u32 lba = (u32)(size_t)L + d * (SLOT / 2), lbb = lba + 65536 - 57472;
        u32x2 gr = __builtin_amdgcn_raw_buffer_load_b64(rloc, gvo, (u32)d * (NBLK * GSTRIDE), SYS);
        CYC(u64 c_poll = 0; u64 c_ld = 0; u64 c_free = 0; u64 c_dec = 0;)
#pragma unroll 1
        for (int s = MODE(2) ? 120 + d : d; s < NKS; s += 8) {
            CYC(const u64 c0 = cyc();)
            if (((u32)__ballot(gr[1] >= call) & 15u) != 15u && !MODE(1)) {
                Spin sp;
                while (true) {
                    gr = __builtin_amdgcn_raw_buffer_load_b64(rloc, gvo, (u32)s * (NBLK * GSTRIDE), SYS);
                    if (((u32)__ballot(gr[1] >= call) & 15u) == 15u) break;
                    if (sp.expired(ABORT)) {
                        *ABORT = 1u;
                        if (lane == 0) __hip_atomic_store((GAS u32*)(myws + OFF_TMO), call, __ATOMIC_RELAXED, __HIP_MEMORY_SCOPE_AGENT);
                        break;
                    }
                    POLL_SLEEP();
                }
            }
            CBAR();
            CYC(const u64 c1 = cyc();)
            u32 gl[4];
#pragma unroll
            for (int u = 0; u < 4; ++u) gl[u] = (u32)__builtin_amdgcn_readlane((int)gr[0], u);
            gr = __builtin_amdgcn_raw_buffer_load_b64(rloc, s + 8 < NKS ? gvo : OOB, (u32)(s + 8) * (NBLK * GSTRIDE), SYS);
            u32x4 q[4][4];
#pragma unroll
            for (int u = 0; u < 4; ++u) {
                const u32 blk = u < 2 ? 2 * ti + u : 32 + 2 * tj + (u - 2);
                const u32 so = unit_off((u32)s, blk);
                const bool he = (gl[u] >> (8 + (lane & 15))) & 1u;
                q[u][0] = __builtin_amdgcn_raw_buffer_load_b128(rloc, lo16, so, SYS);
                q[u][1] = __builtin_amdgcn_raw_buffer_load_b128(rloc, lo16 + 1024, so, SYS);
                q[u][2] = __builtin_amdgcn_raw_buffer_load_b128(rloc, lo16 + 2048, so, SYS);
                q[u][3] = __builtin_amdgcn_raw_buffer_load_b128(rloc, he ? lo16 + 3072 : OOB, so, SYS);
            }
            CYC(WAIT_VM(0); const u64 c2 = cyc();)
            if (s >= NSLOT && !MODE(2)) {
                const u32 need = (u32)(s - NSLOT + 1);
                while (true) {
                    const u32x4 pr = *(volatile LAS u32x4*)(L + L_PROG);
                    if (min(min(pr[0], pr[1]), min(pr[2], pr[3])) >= need) break;
                    __builtin_amdgcn_s_sleep(1);
                }
            }
            CBAR();
            CYC(const u64 c3 = cyc();)
#define DEC(u, o, base) decode_store<o>(q[u][0], q[u][1], q[u][2], q[u][3], (0x80u + (gl[u] & 0x7fu)) * 0x01010101u, base)
            DEC(0, 0, lba); DEC(1, 4096, lba); DEC(2, 57472, lbb); DEC(3, 57472 + 4096, lbb);
            WAIT_LGKM0();
            FULL[d] = (u32)s + 1u;
            CYC(const u64 c4 = cyc(); c_poll += c1 - c0; c_ld += c2 - c1; c_free += c3 - c2; c_dec += c4 - c3;)
        }
        CYC(if (d == 3 && lane == 0) { dbg[gw * 16 + 8] = c_poll; dbg[gw * 16 + 9] = c_ld; dbg[gw * 16 + 10] = c_free; dbg[gw * 16 + 11] = c_dec; })
    }
    WAIT_LGKM0();
    __builtin_amdgcn_s_barrier();
    if (wave < 4) {
        const int wm = wave >> 1, wn = wave & 1;
#pragma unroll
        for (int i = 0; i < 2; ++i) {
            const int row = wm * 64 + i * 32 + (lane & 31);
#pragma unroll
            for (int j = 0; j < 2; ++j)
#pragma unroll
                for (int t = 0; t < 4; ++t) {
                    const int ch = wn * 16 + (2 * (t & 1) + (lane >> 5)) * 4 + 2 * j + (t >> 1);
                    *(LAS f32x4*)(L + row * 512 + ((ch ^ (row & 7)) * 16)) = f32x4{acc[i][j][4 * t], acc[i][j][4 * t + 1], acc[i][j][4 * t + 2], acc[i][j][4 * t + 3]};
                }
        }
        WAIT_LGKM0();
        __builtin_amdgcn_s_barrier();
        return;
    }
    const int d = wave - 4;
    if (d == 0 && lane < NR) {
        const u32x4 wm4 = *(volatile LAS u32x4*)(L + L_WMAX);
        const u32 m = max(max(wm4[0], wm4[1]), max(wm4[2], wm4[3]));
        __hip_atomic_store(amax_dst, ((u64)call << 32) | m, __ATOMIC_RELAXED, __HIP_MEMORY_SCOPE_SYSTEM);
    }
    const int owner = (tj < 4 ? rank : (((rank ^ 4) & 4) | (tj - 4))) & (NR - 1);
    const rsrc_t rout = mk(p.out[owner], KDIM * SHARD);
    GAS u32* const done = (GAS u32*)(p.ws[owner] + OFF_DONE + ((rank & 3) * 16 + ti) * 4);
    u32 ovo;
    {
        const u32 col = tj < 4 ? tj * 128 : ((rank + QROT * tj) & 3) * 128;
        const int blk = 2 * ti + (d >> 2);
        const int srcr = blk < 20 ? ((rank & 4) | (blk / 5)) : (((rank ^ 4) & 4) | ((blk - 20) / 3));
        const int rb = blk < 20 ? blk % 5 : 5 + (blk - 20) % 3;
        ovo = (u32)(phys(srcr) * SHARD + rb * 64 + (d & 3) * 16 + (lane >> 3)) * SHARD + col + (lane & 7) * 16;
    }
    { const u32 wv = __builtin_amdgcn_raw_buffer_load_b32(rout, (lane & 31) == 0 ? ovo + (lane >> 5) * 4 * SHARD : OOB, 0, SYS); asm volatile("" :: "v"(wv)); }
    __builtin_amdgcn_s_barrier();
    f32x4 v[2][4];
#pragma unroll
    for (int h = 0; h < 2; ++h) {
        const int row = d * 16 + h * 8 + (lane >> 3);
#pragma unroll
        for (int j = 0; j < 4; ++j) v[h][j] = *(LAS f32x4*)(L + row * 512 + ((((lane & 7) * 4 + j) ^ (row & 7)) * 16));
    }
    {
        Spin sp;
        u32 mx = 0, gvo = OFF_AMAX + d * 1024 + lane * 16;
        while (true) {
            const u32x4 g2 = __builtin_amdgcn_raw_buffer_load_b128(rloc, gvo, 0, SYS);
            if (min(g2[1], g2[3]) >= call) { mx = max(g2[0], g2[2]); gvo = OOB; }
            if (__all(gvo == OOB)) break;
            if (sp.expired(ABORT)) {
                *ABORT = 1u;
                if (lane == 0) __hip_atomic_store((GAS u32*)(myws + OFF_TMO), call, __ATOMIC_RELAXED, __HIP_MEMORY_SCOPE_AGENT);
                break;
            }
        }
        GMAX[d] = wave_max_u32(mx);
    }
    WAIT_LGKM0();
    __builtin_amdgcn_s_barrier();
    {
        const u32x4 ga = *(volatile LAS u32x4*)(L + L_GMAX), gb = *(volatile LAS u32x4*)(L + L_GMAX + 16);
        const bool bad = *ABORT != 0u;
        const u32 mb = max(max(max(ga[0], ga[1]), max(ga[2], ga[3])), max(max(gb[0], gb[1]), max(gb[2], gb[3])));
        const float amax = __builtin_bit_cast(float, mb);
        const float inv = amax > 0.f ? 127.0f / amax : 0.f;
#pragma unroll
        for (int h = 0; h < 2; ++h) {
            u32x4 o;
#pragma unroll
            for (int r = 0; r < 4; ++r) {
                u32 qb[4];
#pragma unroll
                for (int j = 0; j < 4; ++j) {
                    const float y = __builtin_amdgcn_fmed3f(v[h][j][r] * inv, -127.0f, 127.0f);
                    qb[j] = __builtin_bit_cast(u32, y + 12582912.0f);
                }
                o[r] = __builtin_amdgcn_perm(__builtin_amdgcn_perm(qb[3], qb[2], 0x0c0c0400u), __builtin_amdgcn_perm(qb[1], qb[0], 0x0c0c0400u), 0x05040100u);
                if (bad) o[r] = 0x80808080u;
            }
            __builtin_amdgcn_raw_buffer_store_b128(o, rout, ovo + h * 8 * SHARD, 0, SYS);
        }
    }
    if (tj >= 4) {
        WAIT_VM(0);
        __builtin_amdgcn_s_barrier();
        if (d == 0 && lane == 0) __hip_atomic_store(done, call, __ATOMIC_RELAXED, __HIP_MEMORY_SCOPE_SYSTEM);
    }
}

extern "C" void kernel_launch(int rank, int nranks, ncclComm_t comm, hipStream_t stream,
                              void* const* d_in, const int* in_sizes, int n_in,
                              void* d_out, int out_size, const int* in_dtypes, int out_dtype,
                              void* const* d_out_all, void* const* d_ws_all, size_t ws_bytes) {
    (void)comm; (void)in_dtypes; (void)out_dtype; (void)d_out;
    if (nranks != NR || (unsigned)rank >= (unsigned)NR || n_in < 2 || in_sizes[0] != SHARD * KDIM || in_sizes[1] != SHARD * KDIM ||
        out_size != KDIM * SHARD || ws_bytes < (size_t)WS_USED) {
        fprintf(stderr, "unexpected problem shape\n");
        return;
    }
    Params p;
    p.x = (const float*)d_in[0];
    p.w = (const float*)d_in[1];
    for (int r = 0; r < NR; ++r) {
        p.ws[r] = (char*)d_ws_all[phys(r)];
        p.out[r] = (signed char*)d_out_all[phys(r)];
        if (phys(r) == rank) p.rank = r;
    }
    p.pad = 0;
    HIP_CHECK(hipFuncSetAttribute((const void*)ag_gemm_q8, hipFuncAttributeMaxDynamicSharedMemorySize, LDS_BYTES));
    hipLaunchKernelGGL(ag_gemm_q8, dim3(256), dim3(NTHREAD), LDS_BYTES, stream, p);
    HIP_CHECK(hipGetLastError());
}
```

```cpp
#include <hip/hip_runtime.h>
#include <rccl/rccl.h>
#include <cstdio>

#define HIP_CHECK(expr) do { hipError_t _e = (expr); if (_e != hipSuccess) { \
    fprintf(stderr, "HIP error %s at %s:%d\n", hipGetErrorString(_e), __FILE__, __LINE__); \
    return; } } while (0)

typedef unsigned u32;
typedef unsigned long long u64;
typedef u32 u32x3 __attribute__((ext_vector_type(3)));
typedef u32 u32x4 __attribute__((ext_vector_type(4)));
typedef float f32x2 __attribute__((ext_vector_type(2)));
typedef float f32x4 __attribute__((ext_vector_type(4)));
typedef float f32x16 __attribute__((ext_vector_type(16)));
typedef __bf16 bf16x2 __attribute__((ext_vector_type(2)));
typedef __bf16 bf16x8 __attribute__((ext_vector_type(8)));
typedef __amdgpu_buffer_rsrc_t rsrc_t;
#define LAS __attribute__((address_space(3)))
#define GAS __attribute__((address_space(1)))

constexpr int NR = 8;
constexpr int KDIM = 4096, SHARD = 512;
constexpr int NKS = 128;
constexpr int NBLK = 50;
constexpr u32 UNIT = 3072;
constexpr u32 U_CODE = 2048, U_HDR = 2816;
constexpr int NPW = 80;
constexpr int NARM = 128 - NPW;
constexpr int NUNIT = NKS * 16;
constexpr u32 OFF_CALL = 0, OFF_TMO = 128;
constexpr u32 OFF_TICK = 1024;
constexpr u32 OFF_ARM = 4096;
constexpr u32 OFF_AMAX = 16384;
constexpr u32 OFF_STAGE = 65536;
constexpr u32 STAGE_BYTES = NKS * NBLK * UNIT;
constexpr u32 OFF_RAW = OFF_STAGE + 2 * STAGE_BYTES;
constexpr u32 WS_USED = OFF_RAW + NKS * NBLK * 4096;
constexpr u32 OOB = 0x80000000u;
constexpr int SYS = 17;
constexpr u64 DEADLINE = 40000000ull;

constexpr int NDEC = 12;
constexpr int NWAVE = 4 + NDEC, NTHREAD = NWAVE * 64;
constexpr int NSLOT = 8, SLOT = 16384;
constexpr int L_FULL = NSLOT * SLOT;
constexpr int L_PROG = L_FULL + 32;
constexpr int L_WMAX = L_PROG + 16;
constexpr int L_GMAX = L_WMAX + 16;
constexpr int L_ALLOC = L_GMAX + 32;
constexpr int L_SCALE = L_ALLOC + 16;
constexpr int L_ABORT = L_SCALE + 16;
constexpr int LDS_BYTES = L_ABORT + 32;

struct Params {
    char* ws[NR];
    const float* x;
    const float* w;
    int rank;
    int pad;
    signed char* out[NR];
};

__device__ __forceinline__ int row_of(int blk, int g) {
    return blk < 20 ? ((g & 4) | (blk / 5)) * SHARD + (blk % 5) * 64
                    : (((g ^ 4) & 4) | (blk < 28 ? (blk - 20) >> 1 : blk - 28)) * SHARD + (blk < 28 ? 5 + (blk & 1) : 7) * 64;
}
__device__ __forceinline__ u64 rt() { return __builtin_amdgcn_s_memrealtime(); }
__device__ __forceinline__ rsrc_t mk(const void* p, u32 bytes) {
    return __builtin_amdgcn_make_buffer_rsrc(const_cast<void*>(p), 0, bytes, 0x00020000);
}
__device__ __forceinline__ u32 uni(u32 v) { return __builtin_amdgcn_readfirstlane(v); }
__device__ __forceinline__ u32 cvt2(float a, float b) {
    f32x2 v = {a, b};
    return __builtin_bit_cast(u32, __builtin_convertvector(v, bf16x2));
}
__device__ __forceinline__ u32 bfi(u32 m, u32 a, u32 b) { return (m & a) | (~m & b); }
__device__ __forceinline__ u32 wave_max_u32(u32 v) {
    v = max(v, (u32)__builtin_amdgcn_update_dpp(0, (int)v, 0x111, 0xf, 0xf, true));
    v = max(v, (u32)__builtin_amdgcn_update_dpp(0, (int)v, 0x112, 0xf, 0xf, true));
    v = max(v, (u32)__builtin_amdgcn_update_dpp(0, (int)v, 0x114, 0xf, 0xf, true));
    v = max(v, (u32)__builtin_amdgcn_update_dpp(0, (int)v, 0x118, 0xf, 0xf, true));
    v = max(v, (u32)__builtin_amdgcn_update_dpp(0, (int)v, 0x142, 0xa, 0xf, false));
    v = max(v, (u32)__builtin_amdgcn_update_dpp(0, (int)v, 0x143, 0xc, 0xf, false));
    return (u32)__builtin_amdgcn_readlane((int)v, 63);
}
struct Spin {
    u64 t0 = 0;
    u32 n = 0;
    __device__ __forceinline__ bool expired(volatile LAS u32* abort_word) {
        if ((++n & 63u) != 0u) return false;
        if (n == 64u) t0 = rt();
        return (abort_word && *abort_word != 0u) || rt() - t0 > DEADLINE;
    }
};
#define WAIT_VM(n) asm volatile("s_waitcnt vmcnt(" #n ")" ::: "memory")
#define WAIT_LGKM0() asm volatile("s_waitcnt lgkmcnt(0)" ::: "memory")
#define CBAR() asm volatile("" ::: "memory")
constexpr int KEEP = 12;
constexpr u32 M3 = 0x83838383u;
constexpr u32 H_VALID = 0x8000u, H_LONG = 0x200u, H_RAW = 0x400u, H_NULL = 0xffe0u;
#define ILV2() __builtin_amdgcn_sched_group_barrier(0x8, 1, 0); __builtin_amdgcn_sched_group_barrier(0x100, 2, 0)
#define ILVA() do { ILV2(); ILV2(); ILV2(); __builtin_amdgcn_sched_group_barrier(0x8, 1, 0); } while (0)
#define ILVB() do { ILV2(); ILV2(); __builtin_amdgcn_sched_group_barrier(0x8, 2, 0); } while (0)

__device__ __forceinline__ rsrc_t mk2(u64 base, u32 bytes) { return mk((const void*)base, bytes); }
__device__ __forceinline__ u64 sel64(bool c, u64 a, u64 b) { return b ^ ((a ^ b) & (0ull - (u64)c)); }
__device__ __forceinline__ u32 sel32(bool c, u32 a, u32 b) { return b ^ ((a ^ b) & (0u - (u32)c)); }

__device__ __forceinline__ void push_load(f32x4 (&f)[8], u32 j, u64 px, u64 pw, u32 xvo, u32 wvo) {
    const u32 ks = j >> 4, u = j & 7u;
    const bool ix = (j & 8u) == 0u;
    const rsrc_t r = mk2(sel64(ix, px, pw), SHARD * KDIM * 4);
    const u32 so = sel32(ix, ks * 128u + u * (64u * KDIM * 4u), ks * (32u * SHARD * 4u) + u * 256u);
    const u32 s1 = sel32(ix, 64u, SHARD * 4u), s2 = sel32(ix, 16u * KDIM * 4u, 2u * SHARD * 4u), s4 = sel32(ix, 32u * KDIM * 4u, 16u * SHARD * 4u);
    const u32 vo = (wvo ^ ((xvo ^ wvo) & (0u - (u32)ix))) | (j < NUNIT ? 0u : OOB), vo1 = vo + s1;
#pragma unroll
    for (int i = 0; i < 8; ++i)
        f[i] = __builtin_bit_cast(f32x4, __builtin_amdgcn_raw_buffer_load_b128(r, (i & 1) ? vo1 : vo, so + ((i & 2) ? s2 : 0u) + ((i & 4) ? s4 : 0u), 0));
}

__device__ __forceinline__ u32 pack_params(f32x4 s) {
    float v = (__builtin_fabsf(s[0]) + __builtin_fabsf(s[1])) + (__builtin_fabsf(s[2]) + __builtin_fabsf(s[3]));
#define DADD(ctrl, rm, bc) v += __builtin_bit_cast(float, __builtin_amdgcn_update_dpp(0, __builtin_bit_cast(int, v), ctrl, rm, 0xf, bc))
    DADD(0x111, 0xf, true); DADD(0x112, 0xf, true); DADD(0x114, 0xf, true); DADD(0x118, 0xf, true); DADD(0x142, 0xa, false); DADD(0x143, 0xc, false);
#undef DADD
    const u32 et = ((u32)__builtin_amdgcn_readlane(__builtin_bit_cast(int, v * (3.32f / 256.0f)), 63) >> 23) & 0xffu;
    const u32 a8 = et >= 40u ? 8u - ((et + 1u) & 7u) : 15u - (et & 7u);
    return ((et + a8 - 8u) >> 3) | (a8 << 5);
}

__device__ __forceinline__ u32 push_unit(f32x4 (&f)[8], bool isx, u32 ks,
                                         u32 tbase, u64 px, u64 pw, u32 xvo, u32 wvo, u64 self,
                                         u64 d0, u64 d1, u64 d2, u64 d3, u32 b0, u32 b1, u32 b4, u32 par, u32 ubase, int lane, LAS char* S) {
    const u32 lo16 = (u32)lane * 16u, l0 = lane == 0 ? 0u : OOB;
    const u32 lim = sel32(ks < (u32)NKS, WS_USED, 0u);
    const float sc = __builtin_bit_cast(float, ((par >> 5) + 119u) << 23);
#pragma unroll
    for (int i = 0; i < 8; ++i) f[i] *= sc;
    u32 pk[16];
#define PIN() asm volatile("" : "+v"(pk[0]), "+v"(pk[1]), "+v"(pk[2]), "+v"(pk[3]), "+v"(pk[4]), "+v"(pk[5]), "+v"(pk[6]), "+v"(pk[7]), \
                                "+v"(pk[8]), "+v"(pk[9]), "+v"(pk[10]), "+v"(pk[11]), "+v"(pk[12]), "+v"(pk[13]), "+v"(pk[14]), "+v"(pk[15]))
    if (isx) {
#pragma unroll
        for (int i = 0; i < 16; ++i) {
            const int kh = i >> 3, c = ((i >> 2) & 1) * 2 + (i & 1), e = (i >> 1) & 1;
            pk[i] = cvt2(f[2 * c + kh][2 * e], f[2 * c + kh][2 * e + 1]);
        }
        PIN();
    } else {
#pragma unroll
        for (int i = 0; i < 16; ++i) {
            const int kh = i >> 3, c = ((i >> 2) & 1) * 2 + (i & 1), e = (i >> 1) & 1;
            pk[i] = cvt2(f[4 * kh + 2 * e][c], f[4 * kh + 2 * e + 1][c]);
        }
        PIN();
    }
#undef PIN
    u32 tk = (u32)__builtin_amdgcn_raw_ptr_buffer_atomic_add_i32(1, mk2(self, lim), l0 + OFF_TICK, 0, 0);
    const u32 KU = ((par & 31u) << 2) * 0x01010101u;
    u32x4 lo0, lo1;
    u32 t[8], mk = 0;
    volatile LAS u32* S32 = (volatile LAS u32*)S;
#pragma unroll
    for (int j = 0; j < 8; ++j) {
        const u32 Lb = __builtin_amdgcn_perm(pk[2 * j + 1], pk[2 * j], 0x06040200u);
        t[j] = __builtin_amdgcn_perm(pk[2 * j + 1], pk[2 * j], 0x07050301u);
        const u32 ex = (t[j] ^ KU) & 0x7c7c7c7cu;
        S32[256 + j * 64 + lane] = ex;
        mk = bfi(0x01010101u << j, (ex + 0x7c7c7c7cu) >> (7 - j), mk);
        if (j < 4) lo0[j] = Lb; else lo1[j - 4] = Lb;
    }
    const u32 W0 = bfi(M3, t[0], bfi(0x1c1c1c1cu, __builtin_amdgcn_alignbit(t[1], t[1], 29), t[6] << 5));
    const u32 W1 = bfi(M3, t[2], bfi(0x1c1c1c1cu, __builtin_amdgcn_alignbit(t[3], t[3], 29), t[7] << 5));
    const u32 W2 = bfi(M3, t[4], bfi(0x1c1c1c1cu, __builtin_amdgcn_alignbit(t[5], t[5], 29), bfi(0x20202020u, t[6] >> 2, t[7] >> 1)));
    const rsrc_t r0 = mk2(d0, lim), r1 = mk2(d1, lim), r2 = mk2(d2, sel32(isx, lim, 0u)), r3 = mk2(d3, sel32(isx, lim, 0u));
    const rsrc_t r4 = mk2(self, sel32(b4 != 0u, lim, 0u));
    const u32 i0 = b0 * NKS + ks, i1 = b1 * NKS + ks, i4 = b4 * NKS + ks;
#define PUT(data, vo, so0, so1, so4) do { __builtin_amdgcn_raw_buffer_store_b128(data, r4, vo, so4, SYS); \
                                     __builtin_amdgcn_raw_buffer_store_b128(data, r0, vo, so0, SYS); \
                                     __builtin_amdgcn_raw_buffer_store_b128(data, r1, vo, so1, SYS); \
                                     __builtin_amdgcn_raw_buffer_store_b128(data, r2, vo, so1, SYS); \
                                     __builtin_amdgcn_raw_buffer_store_b128(data, r3, vo, so1, SYS); } while (0)
    const u32 so0 = ubase + i0 * UNIT, so1 = ubase + i1 * UNIT, so4 = ubase + i4 * UNIT;
    PUT(lo0, lo16, so0, so1, so4);
    PUT(lo1, lo16 + 1024, so0, so1, so4);
    volatile LAS unsigned short* SH = (volatile LAS unsigned short*)(S + 768);
    S32[lane * 3] = W0; S32[lane * 3 + 1] = W1; S32[lane * 3 + 2] = W2;
    const u32 mine = (u32)__builtin_popcount(mk);
    u32 inc = mine;
#define DADD(ctrl, rm, bc) inc += (u32)__builtin_amdgcn_update_dpp(0, (int)inc, ctrl, rm, 0xf, bc)
    DADD(0x111, 0xf, true); DADD(0x112, 0xf, true); DADD(0x114, 0xf, true); DADD(0x118, 0xf, true); DADD(0x142, 0xa, false); DADD(0x143, 0xc, false);
#undef DADD
    const u32 cnt = (u32)__builtin_amdgcn_readlane((int)inc, 63);
    const bool raw = cnt > 63u || __ballot(min(min(min(lo0[0], lo1[0]), W0), min(W1, W2)) == 0u) != 0ull;
    const u32 meta = H_VALID | par | (cnt > 31u ? H_LONG : 0u);
    if (!raw) {
        u32 slot = inc - mine + 1u;
        while (mk != 0u) {
            const u32 i = (u32)__builtin_ctz(mk);
            mk &= mk - 1u;
            SH[slot++] = (unsigned short)((i & 7u) * 256u + (u32)lane * 4u + (i >> 3));
        }
        const u32 a = SH[lane] & 0x7ffu;
        const u32 fld = ((volatile LAS unsigned char*)S)[1024u + a];
        const u32 ent = ((((a & 0x700u) | ((a & 2u) << 6) | ((a >> 1) & 0x7eu) | (a & 1u)) << 5) | (fld >> 2));
        SH[lane] = (unsigned short)(lane == 0 ? meta : ((u32)lane <= cnt ? ent : H_NULL));
    }
    u32x4 tail = *(volatile LAS u32x4*)(S + lane * 16);
    u32 tvo = lane < (cnt > 31u ? 56 : 52) ? lo16 + U_CODE : OOB;
    if (__builtin_expect(raw, 0)) {
#pragma unroll
        for (int k = 0; k < 4; ++k) PUT((u32x4{pk[4 * k], pk[4 * k + 1], pk[4 * k + 2], pk[4 * k + 3]}), lo16 + k * 1024, OFF_RAW + i0 * 4096u, OFF_RAW + i1 * 4096u, OFF_RAW + i4 * 4096u);
        tail = u32x4{meta | H_RAW, 0u, 0u, 0u};
        tvo = l0 + U_HDR;
        WAIT_VM(0);
    }
    PUT(tail, tvo, so0, so1, so4);
#undef PUT
    asm volatile("" : "+v"(tk) :: "memory");
    const u32 nn = sel32(ks < (u32)NKS, uni(tk) - tbase, NUNIT);
    push_load(f, nn, px, pw, xvo, wvo);
    __builtin_amdgcn_s_waitcnt(0x0F70 | ((KEEP + 8) & 15) | (((KEEP + 8) >> 4) << 14));
    return nn;
}

__device__ __forceinline__ void push_wave(const Params& p, int rank, u32 v, int lane, LAS char* S) {
    u64 px = (u64)p.x, pw = (u64)p.w;
    u64 w0 = (u64)p.ws[0], w1 = (u64)p.ws[1], w2 = (u64)p.ws[2], w3 = (u64)p.ws[3], w4 = (u64)p.ws[4], w5 = (u64)p.ws[5], w6 = (u64)p.ws[6], w7 = (u64)p.ws[7];
    asm volatile("" : "+s"(px), "+s"(pw), "+s"(w0), "+s"(w1), "+s"(w2), "+s"(w3), "+s"(w4), "+s"(w5), "+s"(w6), "+s"(w7));
    const u32 xvo = ((u32)(lane >> 2) * KDIM + 4u * (lane & 3)) * 4u;
    const u32 wvo = (4u * (lane & 3) * SHARD + 4u * (lane >> 2)) * 4u;
    f32x4 fa[8], fb[8];
    u32 cur = v, nxt = v + NPW;
    u32 callv = __builtin_amdgcn_raw_buffer_load_b32(mk2(w3, WS_USED), OFF_CALL, 0, 16);
    f32x4 sx = __builtin_bit_cast(f32x4, __builtin_amdgcn_raw_buffer_load_b128(mk2(px, SHARD * KDIM * 4), ((u32)(lane >> 2) * (32u * KDIM) + 4u * (lane & 3)) * 4u, 0, 0));
    f32x4 sw = __builtin_bit_cast(f32x4, __builtin_amdgcn_raw_buffer_load_b128(mk2(pw, SHARD * KDIM * 4), ((u32)(lane >> 2) * (256u * SHARD) + 4u * (lane & 3)) * 4u, 0, 0));
    __builtin_amdgcn_sched_barrier(0);
    push_load(fa, cur, px, pw, xvo, wvo);
    asm volatile("" : "+v"(callv), "+v"(sx), "+v"(sw) :: "memory");
    const u32 call = uni(callv) + 1u;
    const u32 parx = pack_params(sx), parw = pack_params(sw);
    push_load(fb, nxt, px, pw, xvo, wvo);
    __builtin_amdgcn_s_waitcnt(0x0F78);
    const u32 tbase = (call - 1u) * (u32)NUNIT - 2u * NPW;
    const u32 ubase = OFF_STAGE + (call & 1u) * STAGE_BYTES;
#define UNIT_STEP(f) do {                                                                                              \
        const u32 u = cur & 7u;                                                                                        \
        const bool isx = (cur & 8u) == 0u, own = u < 5u;                                                               \
        const u32 wr = ((u >> 1) - 2u * rank - 1u) & 3u;       \
        const u64 wd = sel64(wr & 2u, sel64(wr & 1u, w7, w6), sel64(wr & 1u, w5, w4));                                 \
        const u32 bx = sel32(own, (rank & 3) * 5 + u, sel32(u < 7u, 15 + (rank & 3) * 2 + u, 28 + (rank & 3)));         \
        const u32 nn = push_unit(f, isx, cur >> 4, tbase, px, pw, xvo, wvo, w3, sel64(isx, sel64(own, w0, w4), wd),       \
                                 sel64(isx, sel64(own, w1, w5), w3), sel64(own, w2, w6), sel64(own, w3, w7),           \
                                 sel32(isx, bx, 40 + (rank & 3) * 2 + (u & 1u)), sel32(isx, bx, 32 + u),               \
                                 sel32(isx && !own && u < 7u, 43 + u, 0u), sel32(isx, parx, parw), ubase, lane, S);                                              \
        cur = nxt;                                                                                                     \
        nxt = nn;                                                                                                      \
    } while (0)
#pragma unroll 1
    do {
        UNIT_STEP(fa);
        UNIT_STEP(fb);
    } while (cur < (u32)NUNIT);
}

template <int OFF>
__device__ __forceinline__ void put4(u32 d0, u32 d1, u32 d2, u32 d3) {
    asm volatile("ds_write_addtid_b32 %0 offset:%4\n\tds_write_addtid_b32 %1 offset:%5\n\t"
                 "ds_write_addtid_b32 %2 offset:%6\n\tds_write_addtid_b32 %3 offset:%7"
                 :: "v"(d0), "v"(d1), "v"(d2), "v"(d3), "i"(OFF), "i"(OFF + 256), "i"(OFF + 512), "i"(OFF + 768) : "memory");
}
__device__ __forceinline__ void decode(u32 (&d)[16], u32x4 lo0, u32x4 lo1, u32x3 w, u32 K4, u32 m3, u32 m03, u32 m80) {
    u32 H[8];
    H[0] = (w[0] & m3) | K4; H[1] = (__builtin_amdgcn_alignbit(w[0], w[0], 3) & m3) | K4;
    H[2] = (w[1] & m3) | K4; H[3] = (__builtin_amdgcn_alignbit(w[1], w[1], 3) & m3) | K4;
    H[4] = (w[2] & m3) | K4; H[5] = (__builtin_amdgcn_alignbit(w[2], w[2], 3) & m3) | K4;
    u32 h6 = ((w[0] >> 5) & m03) | K4, h7 = ((w[1] >> 5) & m03) | K4;
    asm("" : "+v"(h6), "+v"(h7));
    H[6] = ((w[2] << 2) & m80) | h6;
    H[7] = ((w[2] << 1) & m80) | h7;
#pragma unroll
    for (int j = 0; j < 8; ++j) {
        const u32 l = j < 4 ? lo0[j] : lo1[j - 4];
        d[2 * j] = __builtin_amdgcn_perm(H[j], l, 0x05010400u);
        d[2 * j + 1] = __builtin_amdgcn_perm(H[j], l, 0x07030602u);
    }
}
template <int OFF>
__device__ __forceinline__ void put16(const u32 (&d)[16], u32 addr, u32 fix) {
    put4<OFF>(d[0], d[1], d[2], d[3]);
    put4<OFF + 1024>(d[4], d[5], d[6], d[7]);
    put4<OFF + 2048>(d[8], d[9], d[10], d[11]);
    put4<OFF + 3072>(d[12], d[13], d[14], d[15]);
    if (fix != 0u) asm volatile("ds_xor_b32 %0, %1 offset:%2" :: "v"(addr), "v"(fix), "i"(OFF) : "memory");
}

__global__ void __launch_bounds__(NTHREAD, (NWAVE + 3) / 4) ag_gemm_q8(Params p) {
    extern __shared__ __attribute__((aligned(16))) char smem[];
    LAS char* L = (LAS char*)smem;
    const int tid = threadIdx.x, lane = tid & 63;
    const int wave = (int)uni((u32)tid >> 6);
    const int b = blockIdx.x, rank = p.rank & (NR - 1);
    char* const myws = p.ws[3];
    const rsrc_t rloc = mk(myws, WS_USED);

    if (__builtin_expect(b < NPW, 1)) {
        if (wave == 0) push_wave(p, rank, b, lane, L);
        return;
    }
    if (b < 128) {
        const u32 callv = __builtin_amdgcn_raw_buffer_load_b32(rloc, OFF_CALL, 0, 16);
        const int aw = (b - NPW) * NWAVE + wave;
        const rsrc_t rout = mk(p.out[3], KDIM * SHARD);
        const u32x4 mark = {0x80808080u, 0x80808080u, 0x80808080u, 0x80808080u}, zero = {0u, 0u, 0u, 0u};
        __builtin_amdgcn_s_sleep(127);
        for (int i = aw; i < 1024; i += NARM * NWAVE) {
            __builtin_amdgcn_raw_buffer_store_b128(mark, rout, lane * 16, (u32)row_of(i >> 5, rank ^ 4) * SHARD + (i & 31) * 1024, SYS);
        }
        const u32 call = uni(callv) + 1u;
#pragma unroll 1
        for (int i = aw; i < NKS * NBLK; i += NARM * NWAVE) {
            const u32 so = OFF_STAGE + ((call + 1u) & 1u) * STAGE_BYTES + (u32)i * UNIT;
            __builtin_amdgcn_s_sleep(60);
            __builtin_amdgcn_raw_buffer_store_b128(zero, rloc, lane * 16, so, SYS);
            __builtin_amdgcn_raw_buffer_store_b128(zero, rloc, lane * 16 + 1024, so, SYS);
            __builtin_amdgcn_raw_buffer_store_b128(zero, rloc, lane < 56 ? lane * 16 + 2048 : OOB, so, SYS);
        }
        WAIT_VM(0);
        __builtin_amdgcn_s_barrier();
        if (tid == 0) __hip_atomic_store((GAS u32*)(myws + OFF_ARM + (b - NPW) * 4), call, __ATOMIC_RELAXED, __HIP_MEMORY_SCOPE_AGENT);
        return;
    }

    const int xl = b & 7, li = (b >> 3) - 16;
    const int ti = 4 * (xl >> 1) + (li & 3);
    const int tj = 2 * (xl & 1) + ((li >> 2) & 1) + 4 * (li >> 3);
    const int gw = ti * 8 + tj;
    const bool diag = tj >= 4 && ti == 6 + tj;
    const int ab0 = diag ? 48 : 2 * ti, bb0 = 32 + 2 * (diag ? tj - 4 : tj);
    volatile LAS u32* FULL = (volatile LAS u32*)(L + L_FULL);
    volatile LAS u32* PROG = (volatile LAS u32*)(L + L_PROG);
    volatile LAS u32* WMAX = (volatile LAS u32*)(L + L_WMAX);
    volatile LAS u32* GMAX = (volatile LAS u32*)(L + L_GMAX);
    volatile LAS u32* ABORT = (volatile LAS u32*)(L + L_ABORT);
    if (tid < (L_ABORT + 32 - L_FULL) / 4) ((volatile LAS u32*)(L + L_FULL))[tid] = 0u;
    u32 call = 0;
    WAIT_LGKM0();
    __builtin_amdgcn_s_barrier();
    GAS u64* amax_dst = nullptr;
    if (wave >= 4) {
        call = uni(__hip_atomic_load((GAS u32*)(myws + OFF_CALL), __ATOMIC_RELAXED, __HIP_MEMORY_SCOPE_AGENT)) + 1u;
        amax_dst = (GAS u64*)(p.ws[lane & (NR - 1)] + OFF_AMAX + (rank * 128 + gw) * 8);
        asm volatile("" : "+v"(amax_dst));
    }

    f32x16 acc[2][2];
    if (wave < 4) {
        __builtin_amdgcn_s_setprio(2);
        const int wm = wave >> 1, wn = wave & 1;
        LAS char* pa = L + wm * 4096 + lane * 16;
        LAS char* pb = L + 65536 + wn * 4096 + lane * 16;
#pragma unroll
        for (int i = 0; i < 2; ++i)
#pragma unroll
            for (int j = 0; j < 2; ++j)
#pragma unroll
                for (int r = 0; r < 16; ++r) acc[i][j][r] = 0.f;
        bf16x8 H0[4], H1[4], H2[4];
        auto rd = [&](bf16x8* H, int s, int kh) {
            const int o = (s & 7) * (SLOT / 2) + kh * 2048;
            H[0] = *(LAS bf16x8*)(pa + o); H[2] = *(LAS bf16x8*)(pb + o);
            H[1] = *(LAS bf16x8*)(pa + o + 1024); H[3] = *(LAS bf16x8*)(pb + o + 1024);
        };
        auto mm = [&](bf16x8* H) {
#pragma unroll
            for (int i = 0; i < 2; ++i)
#pragma unroll
                for (int j = 0; j < 2; ++j) acc[i][j] = __builtin_amdgcn_mfma_f32_32x32x16_bf16(H[2 + j], H[i], acc[i][j], 0, 0, 0);
        };
        auto full_of = [&](int s) { return FULL[s & 7]; };
        auto wait_full = [&](u32 seen, int s) {
            while (seen < (u32)s + 1u) seen = full_of(s);
            CBAR();
        };
        wait_full(0u, 0);
        const int unscale = 16 - (int)uni(((volatile LAS u32*)(L + L_SCALE))[wm] + ((volatile LAS u32*)(L + L_SCALE))[2]);
        rd(H0, 0, 0);
        u32 fl = full_of(1);
        rd(H1, 0, 1);
#define STEP(s_, C0, C1, N0, LAST) do {                                                                     \
            if (!LAST) wait_full(fl, (s_) + 1);                                              \
            __builtin_amdgcn_sched_barrier(0);                                                        \
            rd(N0, (s_) + 1, 0);                             \
            fl = full_of((s_) + 2);                                                                   \
            mm(C0);                                                                                   \
            ILVA();                                                                                   \
            __builtin_amdgcn_sched_barrier(0);                                                        \
            asm volatile("s_waitcnt lgkmcnt(5)" ::: "memory");            \
            PROG[wave] = (u32)(s_) + 1u;                                                              \
            __builtin_amdgcn_sched_barrier(0);                                                        \
            rd(C0, (s_) + 1, 1);                                                                      \
            mm(C1);                                                                                   \
            ILVB();                                                                                   \
            __builtin_amdgcn_sched_barrier(0);                                                        \
        } while (0)
#pragma unroll 1
        for (int s = 0; s < NKS - 2; s += 3) {
            STEP(s, H0, H1, H2, false);
            STEP(s + 1, H2, H0, H1, false);
            STEP(s + 2, H1, H2, H0, false);
        }
        STEP(NKS - 2, H0, H1, H2, false);
        STEP(NKS - 1, H2, H0, H1, true);
        float mq[4];
#pragma unroll
        for (int i = 0; i < 4; ++i) {
            mq[i] = fabsf(acc[i >> 1][i & 1][0]);
#pragma unroll
            for (int r = 1; r < 16; ++r) mq[i] = fmaxf(mq[i], fabsf(acc[i >> 1][i & 1][r]));
        }
        const float m = __builtin_amdgcn_ldexpf(fmaxf(fmaxf(mq[0], mq[1]), fmaxf(mq[2], mq[3])), unscale);
        const u32 wmx = wave_max_u32(__builtin_bit_cast(u32, m));
        WMAX[wave] = wmx;
    } else {
        const int d = wave - 4;
        const u32 lo16 = (u32)lane * 16u;
        const u32 ubase = OFF_STAGE + (call & 1u) * STAGE_BYTES;
        const u32 hvo = lane < 8 ? ubase + ((lane & 3) < 2 ? ab0 + (lane & 3) : bb0 - 2 + (lane & 3)) * (NKS * UNIT) + (lane < 4 ? 0u : U_HDR) : OOB;
        auto uidx = [&](int u, int s) { return (u32)((u < 2 ? ab0 + u : bb0 - 2 + u) * NKS + s); };
        auto uoff = [&](int u, int s) { return uni(ubase) + uidx(u, s) * UNIT; };
        auto expire = [&]() {
            *ABORT = 1u;
            if (lane == 0) __hip_atomic_store((GAS u32*)(myws + OFF_TMO), call, __ATOMIC_RELAXED, __HIP_MEMORY_SCOPE_AGENT);
        };
        auto hload = [&](int s) { return __builtin_amdgcn_raw_buffer_load_b32(rloc, s < NKS ? hvo : OOB, (u32)s * UNIT, SYS); };
        auto hready = [&](u32 h) { const u32 m = (u32)__ballot(h != 0u); return ((m | (m >> 4)) & 15u) == 15u; };
        u32 m3 = M3, m03 = 0x03030303u, m80 = 0x80808080u;
        asm volatile("" : "+v"(m3), "+v"(m03), "+v"(m80));
        u32 hc = hload(d);
#pragma unroll 1
        for (int s = d; s < NKS; s += NDEC) {
            if (!hready(hc)) {
                Spin sp;
                while (true) {
                    hc = hload(s);
                    if (hready(hc)) break;
                    if (sp.expired(ABORT)) { expire(); break; }
                }
            }
            CBAR();
            u32 meta[4];
            u32x4 q[8];
            u32x3 w[4];
            u32 ent[4];
#pragma unroll
            for (int u = 0; u < 4; ++u) {
                q[2 * u] = __builtin_amdgcn_raw_buffer_load_b128(rloc, lo16, uoff(u, s), SYS);
                q[2 * u + 1] = __builtin_amdgcn_raw_buffer_load_b128(rloc, lo16 + 1024, uoff(u, s), SYS);
                w[u] = __builtin_amdgcn_raw_buffer_load_b96(rloc, (u32)lane * 12u + U_CODE, uoff(u, s), SYS);
                ent[u] = __builtin_amdgcn_raw_buffer_load_b16(rloc, (u32)lane * 2u + U_HDR, uoff(u, s), SYS);
            }
            auto missing = [&]() {
                u32 z = ~0u;
#pragma unroll
                for (int u = 0; u < 4; ++u) {
                    meta[u] = (u32)__builtin_amdgcn_readlane((int)ent[u], 0);
                    const u32 short_hdr = (lane >= 32 && !(meta[u] & H_LONG)) ? 1u : 0u;
                    const u32 zu = min(min(min(q[2 * u][0], q[2 * u + 1][0]), ent[u] | short_hdr), min(min(w[u][0], w[u][1]), w[u][2]));
                    z = min(z, (meta[u] & H_RAW) ? ~0u : zu);
                }
                return __ballot(z == 0u) != 0ull;
            };
            if (missing()) {
                Spin sp;
                do {
#pragma unroll
                    for (int u = 0; u < 4; ++u) {
                        const u32x4 a = __builtin_amdgcn_raw_buffer_load_b128(rloc, q[2 * u][0] == 0u ? lo16 : OOB, uoff(u, s), SYS);
                        const u32x4 c = __builtin_amdgcn_raw_buffer_load_b128(rloc, q[2 * u + 1][0] == 0u ? lo16 + 1024 : OOB, uoff(u, s), SYS);
                        const u32x3 g = __builtin_amdgcn_raw_buffer_load_b96(rloc, min(min(w[u][0], w[u][1]), w[u][2]) == 0u ? (u32)lane * 12u + U_CODE : OOB, uoff(u, s), SYS);
                        const u32 h = __builtin_amdgcn_raw_buffer_load_b16(rloc, ent[u] == 0u ? (u32)lane * 2u + U_HDR : OOB, uoff(u, s), SYS);
                        q[2 * u] |= a; q[2 * u + 1] |= c; w[u] |= g; ent[u] |= h;
                    }
                    if (sp.expired(ABORT)) { expire(); break; }
                } while (missing());
            }
            const u32 hn = hload(s + NDEC);
            u32 dd[4][16], fix[4];
            auto unpack = [&](int u) {
                fix[u] = lane != 0 ? (ent[u] & 31u) << (10u + ((ent[u] >> 1) & 16u)) : 0u;
                if (__builtin_expect(meta[u] & H_RAW, 0)) {
#pragma unroll
                    for (int k = 0; k < 4; ++k) {
                        const u32x4 rw = __builtin_amdgcn_raw_buffer_load_b128(rloc, lo16 + k * 1024, OFF_RAW + uidx(u, s) * 4096u, SYS);
                        dd[u][4 * k] = rw[0]; dd[u][4 * k + 1] = rw[1]; dd[u][4 * k + 2] = rw[2]; dd[u][4 * k + 3] = rw[3];
                    }
                    fix[u] = 0u;
                    __builtin_amdgcn_s_waitcnt(0x0F70);
                } else {
                    decode(dd[u], q[2 * u], q[2 * u + 1], w[u], ((meta[u] & 31u) << 2) * 0x01010101u, m3, m03, m80);
                }
            };
            __builtin_amdgcn_s_setprio(1);
#pragma unroll
            for (int u = 0; u < 4; ++u) { unpack(u); __builtin_amdgcn_sched_barrier(0); }
            u32 old;
            u32x4 pr;
            asm volatile("s_mov_b64 exec, 1\n\tds_add_rtn_u32 %0, %2, %3\n\ts_mov_b64 exec, -1\n\tds_read_b128 %1, %4\n\ts_waitcnt lgkmcnt(0)"
                         : "=&v"(old), "=&v"(pr) : "v"((u32)(size_t)L + L_ALLOC), "v"(1u), "v"((u32)(size_t)L + L_PROG) : "memory");
            const u32 pos = uni(old);
            while (min(min(pr[0], pr[1]), min(pr[2], pr[3])) + (NSLOT - 1) < pos) {
                __builtin_amdgcn_s_sleep(2);
                pr = *(volatile LAS u32x4*)(L + L_PROG);
            }
            CBAR();
            __builtin_amdgcn_s_setprio(3);
            const u32 lba = (u32)(size_t)L + (pos & 7u) * (SLOT / 2), lbb = lba + 65536 - 57472;
#define DEC(u, o, base) do {                                                                  \
                if (!(u & 1)) asm volatile("s_mov_b32 m0, %0\n\ts_nop 0" :: "s"(base));       \
                put16<o>(dd[u], base + ((ent[u] >> 4) & 0xffcu), fix[u]);                     \
            } while (0)
            DEC(0, 0, lba); DEC(1, 4096, lba); DEC(2, 57472, lbb); DEC(3, 57472 + 4096, lbb);
            if (s == d) *(volatile LAS u32x4*)(L + L_SCALE) = u32x4{(meta[0] >> 5) & 15u, (meta[1] >> 5) & 15u, (meta[2] >> 5) & 15u, 0u};
            WAIT_LGKM0();
            FULL[pos & 7u] = pos + 1u;
            __builtin_amdgcn_s_setprio(0);
            hc = hn;
        }
        if (d == 0) {
            Spin sp;
            while (uni(__hip_atomic_load((GAS u32*)(myws + OFF_ARM + (gw % NARM) * 4), __ATOMIC_RELAXED, __HIP_MEMORY_SCOPE_AGENT)) < call)
                if (sp.expired(ABORT)) { expire(); break; }
        }
    }
    WAIT_LGKM0();
    __builtin_amdgcn_s_barrier();
    if (wave < 4) {
        const int wm = wave >> 1, wn = wave & 1;
#pragma unroll
        for (int i = 0; i < 2; ++i) {
            const int row = wm * 64 + i * 32 + (lane & 31);
#pragma unroll
            for (int j = 0; j < 2; ++j)
#pragma unroll
                for (int t = 0; t < 4; ++t) {
                    const int ch = wn * 16 + (2 * (t & 1) + (lane >> 5)) * 4 + 2 * j + (t >> 1);
                    *(LAS f32x4*)(L + row * 512 + ((ch ^ (row & 7)) * 16)) = f32x4{acc[i][j][4 * t], acc[i][j][4 * t + 1], acc[i][j][4 * t + 2], acc[i][j][4 * t + 3]};
                }
        }
        WAIT_LGKM0();
        __builtin_amdgcn_s_barrier();
        return;
    }
    if (wave >= 12) return;
    const int d = wave - 4;
    if (d == 0 && lane < NR) {
        const u32x4 wm4 = *(volatile LAS u32x4*)(L + L_WMAX);
        const u32 m = max(max(wm4[0], wm4[1]), max(wm4[2], wm4[3]));
        __hip_atomic_store(amax_dst, ((u64)call << 32) | m, __ATOMIC_RELAXED, __HIP_MEMORY_SCOPE_SYSTEM);
    }
    const int owner = tj < 4 || diag ? 3 : 4 + ((tj - rank - 1) & 3);
    const rsrc_t rout = mk(p.out[owner], KDIM * SHARD);
    u32 ovo;
    {
        const u32 col = tj < 4 || diag ? (tj & 3) * 128 : ((rank + tj) & 3) * 128;
        const int row0 = diag ? rank * SHARD + (5 + (d >> 2)) * 64 : row_of(2 * ti + (d >> 2), rank);
        ovo = (u32)(row0 + (d & 3) * 16 + (lane >> 3)) * SHARD + col + (lane & 7) * 16;
    }
    {
        const u32 wv = __builtin_amdgcn_raw_buffer_load_b32(rout, (lane & 31) == 0 ? ovo + (lane >> 5) * 4 * SHARD : OOB, 0, SYS);
        asm volatile("" :: "v"(wv));
    }
    __builtin_amdgcn_s_barrier();
    f32x4 v[2][4];
#pragma unroll
    for (int h = 0; h < 2; ++h) {
        const int row = d * 16 + h * 8 + (lane >> 3);
#pragma unroll
        for (int j = 0; j < 4; ++j) v[h][j] = *(LAS f32x4*)(L + row * 512 + ((((lane & 7) * 4 + j) ^ (row & 7)) * 16));
    }
    const int unscale = 16 - (int)(((volatile LAS u32*)(L + L_SCALE))[d >> 2] + ((volatile LAS u32*)(L + L_SCALE))[2]);
    {
        Spin sp;
        u32 mx = 0, gvo = OFF_AMAX + d * 1024 + lane * 16;
        while (true) {
            const u32x4 g = __builtin_amdgcn_raw_buffer_load_b128(rloc, gvo, 0, SYS);
            if (min(g[1], g[3]) >= call) { mx = max(g[0], g[2]); gvo = OOB; }
            if (__all(gvo == OOB)) break;
            if (sp.expired(ABORT)) {
                *ABORT = 1u;
                if (lane == 0) __hip_atomic_store((GAS u32*)(myws + OFF_TMO), call, __ATOMIC_RELAXED, __HIP_MEMORY_SCOPE_AGENT);
                break;
            }
            __builtin_amdgcn_s_sleep(2);
        }
        GMAX[d] = wave_max_u32(mx);
    }
    WAIT_LGKM0();
    __builtin_amdgcn_s_barrier();
    const u32x4 ga = *(volatile LAS u32x4*)(L + L_GMAX), gb = *(volatile LAS u32x4*)(L + L_GMAX + 16);
    const bool bad = uni(*ABORT) != 0u;
    const float amax = __builtin_bit_cast(float, max(max(max(ga[0], ga[1]), max(ga[2], ga[3])), max(max(gb[0], gb[1]), max(gb[2], gb[3]))));
    float inv = 127.0f * __builtin_amdgcn_rcpf(amax);
    if (!(amax > 1e-30f && amax < 1e30f)) inv = amax > 0.f ? 127.0f / amax : 0.f;
    const float mul = __builtin_amdgcn_ldexpf(inv, unscale);
#pragma unroll
    for (int h = 0; h < 2; ++h) {
        u32x4 o;
#pragma unroll
        for (int r = 0; r < 4; ++r) {
            u32 qb[4];
#pragma unroll
            for (int j = 0; j < 4; ++j) qb[j] = __builtin_bit_cast(u32, __builtin_fmaf(v[h][j][r], mul, 12582912.0f));
            o[r] = __builtin_amdgcn_perm(__builtin_amdgcn_perm(qb[3], qb[2], 0x0c0c0400u), __builtin_amdgcn_perm(qb[1], qb[0], 0x0c0c0400u), 0x05040100u);
        }
        if (bad) o = u32x4{0x80808080u, 0x80808080u, 0x80808080u, 0x80808080u};
        __builtin_amdgcn_raw_buffer_store_b128(o, rout, ovo + h * 8 * SHARD, 0, SYS);
        __builtin_amdgcn_sched_barrier(0);
    }
    if (gw == 0 && d == 0 && lane == 0)
        __hip_atomic_store((GAS u32*)(myws + OFF_CALL), call, __ATOMIC_RELAXED, __HIP_MEMORY_SCOPE_AGENT);
    if (tj < 4 && ti != 10 + (rank & 3)) {
        const u32 fvo = (u32)(row_of(2 * ti + (d >> 2), rank ^ 4) + (d & 3) * 16 + (lane >> 3)) * SHARD + ((tj + rank) & 3) * 128 + (lane & 7) * 16;
        Spin sp;
        bool late = false;
        {
            u32 v0 = fvo, v1 = fvo + 8 * SHARD;
            __builtin_amdgcn_s_sleep(80);
            while (true) {
                const u32x4 a = __builtin_amdgcn_raw_buffer_load_b128(rout, v0, 0, SYS), c = __builtin_amdgcn_raw_buffer_load_b128(rout, v1, 0, SYS);
                if (a[0] != 0x80808080u && a[1] != 0x80808080u && a[2] != 0x80808080u && a[3] != 0x80808080u) v0 = OOB;
                if (c[0] != 0x80808080u && c[1] != 0x80808080u && c[2] != 0x80808080u && c[3] != 0x80808080u) v1 = OOB;
                if (__all((v0 & v1) == OOB)) break;
                if (sp.expired(ABORT)) { late = true; break; }
            }
        }
        if (late && lane == 0) __hip_atomic_store((GAS u32*)(myws + OFF_TMO), call, __ATOMIC_RELAXED, __HIP_MEMORY_SCOPE_AGENT);
    }
}

extern "C" void kernel_launch(int rank, int nranks, ncclComm_t comm, hipStream_t stream,
                              void* const* d_in, const int* in_sizes, int n_in,
                              void* d_out, int out_size, const int* in_dtypes, int out_dtype,
                              void* const* d_out_all, void* const* d_ws_all, size_t ws_bytes) {
    (void)comm; (void)in_dtypes; (void)out_dtype; (void)d_out;
    if (nranks != NR || (unsigned)rank >= (unsigned)NR || n_in < 2 || in_sizes[0] != SHARD * KDIM || in_sizes[1] != SHARD * KDIM ||
        out_size != KDIM * SHARD || ws_bytes < (size_t)WS_USED) {
        fprintf(stderr, "unexpected problem shape\n");
        return;
    }
    Params p;
    p.x = (const float*)d_in[0];
    p.w = (const float*)d_in[1];
    for (int r = 0; r < NR; ++r) {
        const int q = ((r < 4 ? rank : rank ^ 4) & 4) | ((rank + 1 + r) & 3);
        p.ws[r] = (char*)d_ws_all[q];
        p.out[r] = (signed char*)d_out_all[q];
    }
    p.rank = rank;
    p.pad = 0;
    HIP_CHECK(hipFuncSetAttribute((const void*)ag_gemm_q8, hipFuncAttributeMaxDynamicSharedMemorySize, LDS_BYTES));
    hipLaunchKernelGGL(ag_gemm_q8, dim3(256), dim3(NTHREAD), LDS_BYTES, stream, p);
    HIP_CHECK(hipGetLastError());
}
```
